```python
import numpy as np

kernel_name = "gpu_a2a_gemm_m4096_k4096_n16384_e4m3e4m3_dequant"


ABSMAX_RTOL = 0.03

COMPUTE_DTYPE = "float32"
EPILOGUE = "dequant"
IN_QUANT = None
OUT_QUANT = None
IN_FP8 = "e4m3e4m3"
IN_DTYPES = {"x": "e4m3", "w_mat": "e4m3",
             "scale_x": "float32", "scale_w": "float32"}
WORLD_SIZE = 8

def _snap_fp8(x, mant_bits, max_abs, min_normal):
    a = np.abs(x.astype(np.float64))
    _m, _e = np.frexp(a)
    q = np.where(a >= min_normal, np.ldexp(1.0, _e - 1 - mant_bits),
                 min_normal * 2.0 ** -mant_bits)
    snapped = np.minimum(np.round(a / q) * q, max_abs)
    return (np.sign(x) * snapped).astype(np.float32)

def setup_inputs():

    x = _snap_fp8(np.random.randn(4096, 4096).astype(np.float32), 3, 448.0, 2.0 ** -6)
    w_mat = _snap_fp8(np.random.randn(4096, 16384).astype(np.float32), 3, 448.0, 2.0 ** -6)
    scale_x = (0.02 * (1.0 + np.random.rand(1))).astype(np.float32)
    scale_w = (0.02 * (1.0 + np.random.rand(1))).astype(np.float32)
    return {"x": x, "w_mat": w_mat, "scale_x": scale_x, "scale_w": scale_w}

def reference(x, w_mat, scale_x, scale_w):
    acc = (x @ w_mat).astype(np.float32)
    y = (acc * (scale_x * scale_w)).astype(np.float32)
    return y
```

```cpp
#include <hip/hip_runtime.h>
#include <rccl/rccl.h>
#include <cstdio>

#define HIP_CHECK(expr) do { hipError_t _e = (expr); if (_e != hipSuccess) { \
    fprintf(stderr, "HIP error %s at %s:%d\n", hipGetErrorString(_e), __FILE__, __LINE__); \
    return; } } while (0)

typedef int v2i __attribute__((ext_vector_type(2)));
typedef int v4i __attribute__((ext_vector_type(4)));
typedef unsigned v4u __attribute__((ext_vector_type(4)));
typedef int v8i __attribute__((ext_vector_type(8)));
typedef float v16f __attribute__((ext_vector_type(16)));
#define LAS __attribute__((address_space(3)))
#define GAS __attribute__((address_space(1)))

constexpr int NRANK = 8;
constexpr int ML = 512;
constexpr int NN = 16384;
constexpr int KSL = 512;
constexpr int NSTEP = 32;
constexpr int NREM = 28;
constexpr int DEPTH = 8;
constexpr int STAGE = 16384;
constexpr int CTRL = DEPTH * STAGE;
constexpr int LDS_BYTES = CTRL + 64;
constexpr unsigned BIG = 64;

constexpr unsigned XSTEP = 65536;
constexpr unsigned XSLOT = (NREM + 2) * XSTEP;
constexpr unsigned FLAG_OFF = 4u << 20;
constexpr int REP = 8;
constexpr unsigned CNTP_OFF = FLAG_OFF + 65536;
constexpr unsigned CNTQ_OFF = CNTP_OFF + 32768;
constexpr unsigned TMO_OFF = CNTQ_OFF + 32768;
constexpr unsigned DBG_OFF = TMO_OFF + 4096;
constexpr unsigned WS_NEED = DBG_OFF + (1u << 20);
constexpr unsigned long long DEADLINE_TICKS = 50000000ull;
#define XAUX 17
#define INFL 4
#define PACE_TICKS 100

struct Args {
    const unsigned char* x;
    const unsigned char* w;
    const float* sx;
    const float* sw;
    float* out;
    unsigned char* ws[NRANK];
    int rank;
    int pad;
};

#define SB() __builtin_amdgcn_sched_barrier(0)
#define CFENCE() asm volatile("" ::: "memory")

#define STAMP(slot) do {} while (0)

__device__ __forceinline__ v8i cat8(v4i lo, v4i hi) {
    return __builtin_shufflevector(lo, hi, 0, 1, 2, 3, 4, 5, 6, 7);
}

__global__ void __launch_bounds__(512, 2) a2a_gemm(Args a) {
    extern __shared__ __attribute__((aligned(16))) unsigned char smem[];
    LAS unsigned char* L = (LAS unsigned char*)smem;
    const int tid = threadIdx.x;
    const int lane = tid & 63;
    const int wave = __builtin_amdgcn_readfirstlane(tid >> 6);
    const int b = blockIdx.x;
    const int rank = a.rank;
    const int idx = b >> 3;
    const int nt = (b & 7) * 16 + (idx >> 1);
    const int mh = idx & 1;
    const int n0 = nt * 128;
    unsigned char* wsl = a.ws[rank];
    STAMP(0);

    if (wave < 4) {
        __builtin_amdgcn_s_setprio(2);
        const int mb = mh * 4 + wave;
        const unsigned char* xloc = a.x + (size_t)(rank * ML + mb * 64) * KSL;
        v8i xf[4][2];
        v8i wf[4];
        v16f acc[2][4];
#pragma unroll
        for (int i = 0; i < 2; ++i)
#pragma unroll
            for (int j = 0; j < 4; ++j)
#pragma unroll
                for (int r = 0; r < 16; ++r) acc[i][j][r] = 0.f;
        __amdgpu_buffer_rsrc_t rx = __builtin_amdgcn_make_buffer_rsrc((void*)xloc, 0, 64 * KSL, 0x00020000);
        unsigned voff = (lane & 31) * KSL + (lane >> 5) * 16;
        unsigned xs = 0, sA = 32 * KSL, sB = 32, inc0 = 64, inc1 = 64;
        __builtin_amdgcn_s_barrier();
        CFENCE();
#pragma unroll
        for (int u = 0; u < 3; ++u)
#pragma unroll
            for (int i = 0; i < 2; ++i) {
                v4i lo = __builtin_amdgcn_raw_buffer_load_b128(rx, voff, u * 64 + i * 32 * KSL, XAUX);
                v4i hi = __builtin_amdgcn_raw_buffer_load_b128(rx, voff, u * 64 + i * 32 * KSL + 32, XAUX);
                xf[u][i] = cat8(lo, hi);
            }
        xs = 3 * 64;
        unsigned wadr[4];
        {
            const int qq = (lane & 15) >> 1, p = lane & 1, cg = (lane >> 4) & 1, h = lane >> 5;
#pragma unroll
            for (int j = 0; j < 4; ++j)
                wadr[j] = (16 * h + qq) * 128 + ((j ^ ((qq >> 1) & 3)) << 5) + 16 * cg + 8 * p;
        }
        volatile LAS v4u* ctrl = (volatile LAS v4u*)(L + CTRL);
        volatile LAS unsigned* done = (volatile LAS unsigned*)(L + CTRL + 16) + wave;
        {
            v4u c = *ctrl;
            while (__builtin_amdgcn_readfirstlane(min(c.x, c.y)) < 1u) { __builtin_amdgcn_s_sleep(1); c = *ctrl; }
        }
        CFENCE();
        STAMP(1);
        auto wread = [&](int j, unsigned stage_off, int kk) -> v8i {
            LAS unsigned char* p = L + stage_off + wadr[j] + kk * 8192;
            v2i r0 = __builtin_amdgcn_ds_read_tr8_b64_v2i32((LAS v2i*)(p));
            v2i r1 = __builtin_amdgcn_ds_read_tr8_b64_v2i32((LAS v2i*)(p + 1024));
            v2i r2 = __builtin_amdgcn_ds_read_tr8_b64_v2i32((LAS v2i*)(p + 4096));
            v2i r3 = __builtin_amdgcn_ds_read_tr8_b64_v2i32((LAS v2i*)(p + 5120));
            return cat8(__builtin_shufflevector(r0, r1, 0, 1, 2, 3), __builtin_shufflevector(r2, r3, 0, 1, 2, 3));
        };
        wf[0] = wread(0, 0, 0);
        wf[1] = wread(1, 0, 0);
        wf[2] = wread(2, 0, 0);
        wf[3] = wf[2];
        xf[3][0] = xf[2][0]; xf[3][1] = xf[2][1];
        SB();
        for (int it = 0; it < NSTEP / 2; ++it) {
            const unsigned so0 = ((2 * it) & (DEPTH - 1)) * STAGE;
            const unsigned so2 = ((2 * it + 2) & (DEPTH - 1)) * STAGE;
            v4u cw;
#pragma unroll
            for (int u = 0; u < 4; ++u) {
                v4i xp[4];
#pragma unroll
                for (int j = 0; j < 4; ++j) {
                    const int s3 = 4 * u + j + 3;
                    const int u3 = s3 >> 2, j3 = s3 & 3;
                    if ((u & 1) == 1 && j == 0) {
                        const unsigned T = 2 * it + (u >> 1);
                        unsigned m = __builtin_amdgcn_readfirstlane(min(min(cw.x, cw.y), cw.z));
                        if (__builtin_expect(m <= T + 1 || T == 2, 0)) {
                            while (m <= T + 1) {
                                __builtin_amdgcn_s_sleep(1);
                                cw = *ctrl;
                                m = __builtin_amdgcn_readfirstlane(min(min(cw.x, cw.y), cw.z));
                            }
                            if (T == 2) {
                                rx = __builtin_amdgcn_make_buffer_rsrc((void*)wsl, 0, 2 * XSLOT, 0x00020000);
                                voff = lane * 16;
                                xs = __builtin_amdgcn_readfirstlane(cw.w) + mb * 8192;
                                sA = 2048; sB = 1024; inc0 = 4096; inc1 = XSTEP - 4096;
                            }
                        }
                        CFENCE();
                    }
                    acc[0][j] = __builtin_amdgcn_mfma_scale_f32_32x32x64_f8f6f4(xf[u][0], wf[j], acc[0][j], 0, 0, 0, 0, 0, 0);
                    acc[1][j] = __builtin_amdgcn_mfma_scale_f32_32x32x64_f8f6f4(xf[u][1], wf[j], acc[1][j], 0, 0, 0, 0, 0, 0);
                    wf[j3] = wread(j3, u3 < 4 ? so0 + (u3 >> 1) * STAGE : so2, u3 & 1);
                    xp[j] = __builtin_amdgcn_raw_buffer_load_b128(rx, voff, xs + (j >> 1) * sA + (j & 1) * sB, XAUX);
                    if (j == 1) xf[(u + 3) & 3][0] = cat8(xp[0], xp[1]);
                    if (j == 3) xf[(u + 3) & 3][1] = cat8(xp[2], xp[3]);
                    if (j == 3) xs += (u & 1) ? inc0 : inc1;
                    __builtin_amdgcn_sched_group_barrier(0x008, 1, 0);
                    __builtin_amdgcn_sched_group_barrier(0x100, 2, 0);
                    __builtin_amdgcn_sched_group_barrier(0x020, 1, 0);
                    __builtin_amdgcn_sched_group_barrier(0x008, 1, 0);
                    __builtin_amdgcn_sched_group_barrier(0x100, 2, 0);
                    SB();
                    if ((u & 1) == 0 && j == 1) { cw = *ctrl; SB(); }
                    if ((u & 1) == 1 && j == 0) {
                        CFENCE();
                        *done = 2 * it + (u >> 1) + 1;
                        CFENCE();
                        SB();
                    }
                }
            }
        }
        STAMP(2);
        const unsigned bad = *(volatile LAS unsigned*)(L + CTRL + 32);
        const float scale = a.sx[0] * a.sw[0];
        const float sc = bad ? __builtin_nanf("") : scale;
        float* o = a.out + (size_t)(mb * 64 + 4 * (lane >> 5)) * NN + n0 + (lane & 31);
#pragma unroll
        for (int i = 0; i < 2; ++i)
#pragma unroll
            for (int r = 0; r < 16; ++r)
#pragma unroll
                for (int j = 0; j < 4; ++j)
                    o[(size_t)(32 * i + (r & 3) + 8 * (r >> 2)) * NN + 32 * j] = acc[i][j][r] * sc;
        STAMP(3);
    } else if (wave < 6) {
        const int ld = wave - 4;
        const int rr = lane >> 3, p = lane & 7;
        const unsigned voff = rr * NN + ((((p >> 1) ^ ((rr >> 1) & 3)) << 5) | ((p & 1) << 4));
        v4i rs;
        {
            const unsigned long long wb = (unsigned long long)(a.w + n0);
            rs.x = __builtin_amdgcn_readfirstlane((int)(unsigned)wb);
            rs.y = __builtin_amdgcn_readfirstlane((int)(unsigned)(wb >> 32) & 0xffff);
            rs.z = 4096 * NN - n0;
            rs.w = 0x00020000;
        }
        const unsigned lbase = (unsigned)(size_t)L + ld * 8192;
        auto issue = [&](int s) {
            unsigned kb;
            if (s < 4) kb = rank * KSL + s * 128;
            else { const int t = s - 4, c = t / 7, i = t - c * 7; kb = ((rank + 1 + i) & 7) * KSL + c * 128; }
            const unsigned so = (kb + ld * 64) * NN;
            const unsigned m = lbase + (s & (DEPTH - 1)) * STAGE;
#pragma unroll
            for (int q = 0; q < 8; ++q)
                asm volatile("s_mov_b32 m0, %0\n\ts_nop 0\n\tbuffer_load_dwordx4 %1, %2, %3 offen lds"
                             :: "s"(m + q * 1024), "v"(voff), "s"(rs), "s"(so + q * 8 * NN) : "memory");
        };
        asm volatile("s_nop 4" ::: "memory");
        __builtin_amdgcn_s_barrier();
        CFENCE();
        volatile LAS unsigned* full = (volatile LAS unsigned*)(L + CTRL) + ld;
        volatile LAS v4u* done = (volatile LAS v4u*)(L + CTRL + 16);
        int issued = 0, pub = 0;
        while (pub < NSTEP) {
            if (issued < NSTEP && issued - pub < (pub == 0 ? 2 : INFL)) {
                const v4u d = *done;
                const int fr = __builtin_amdgcn_readfirstlane(min(min(d.x, d.y), min(d.z, d.w))) + DEPTH;
                CFENCE();
                if (issued < fr) { issue(issued); ++issued; continue; }
            }
            const int n = issued - pub;
            if (n == 0) { __builtin_amdgcn_s_sleep(2); continue; }
            switch (n) {
                case 1: asm volatile("s_waitcnt vmcnt(0)" ::: "memory"); break;
                case 2: asm volatile("s_waitcnt vmcnt(8)" ::: "memory"); break;
                case 3: asm volatile("s_waitcnt vmcnt(16)" ::: "memory"); break;
                case 4: asm volatile("s_waitcnt vmcnt(24)" ::: "memory"); break;
                case 5: asm volatile("s_waitcnt vmcnt(32)" ::: "memory"); break;
                case 6: asm volatile("s_waitcnt vmcnt(40)" ::: "memory"); break;
                default: asm volatile("s_waitcnt vmcnt(48)" ::: "memory"); break;
            }
            ++pub;
            *full = pub < NSTEP ? (unsigned)pub : BIG;
        }
        STAMP(1);
    } else if (wave == 6) {
        const unsigned d = (unsigned)(b >> 5);
        const int mb = (b >> 2) & 7, c = b & 3;
        const bool act = d < (unsigned)NRANK && (int)d != rank;
        GAS unsigned* cnt = (GAS unsigned*)(wsl + CNTP_OFF) + b * 32;
        v4u v[8];
        unsigned call = 0;
        const unsigned long long t0 = __builtin_amdgcn_s_memrealtime();
        if (act) {
            call = __hip_atomic_load(cnt, __ATOMIC_RELAXED, __HIP_MEMORY_SCOPE_AGENT) + 1;
            const GAS unsigned char* src = (const GAS unsigned char*)a.x + (size_t)(d * ML + mb * 64 + (lane & 31)) * KSL
                                           + c * 128 + (lane >> 5) * 16;
#pragma unroll
            for (int q = 0; q < 8; ++q)
                v[q] = *(const GAS v4u*)(src + ((q >> 1) & 1) * 32 * KSL + (q >> 2) * 64 + (q & 1) * 32);
        }
        __builtin_amdgcn_s_barrier();
        if (act) {
            const int t = 7 * c + ((rank - (int)d - 1) & 7);
            unsigned char* pw = a.ws[d];
            unsigned char* dst = pw + (call & 1) * XSLOT + t * XSTEP + mb * 8192 + lane * 16;
            asm volatile("s_waitcnt vmcnt(0)" ::: "memory");
            STAMP(1);
            while (__builtin_amdgcn_s_memrealtime() - t0 < (unsigned long long)(c * PACE_TICKS)) __builtin_amdgcn_s_sleep(2);
#pragma unroll
            for (int q = 0; q < 8; ++q)
                asm volatile("global_store_dwordx4 %0, %1, off sc0 sc1\n\ts_nop 1" :: "v"(dst + q * 1024), "v"(v[q]) : "memory");
            asm volatile("s_waitcnt vmcnt(0)" ::: "memory");
            STAMP(2);
            if (lane < REP) {
                GAS unsigned* f = (GAS unsigned*)(pw + FLAG_OFF + lane * 1024 + (mb >> 2) * 512 + t * 16 + (mb & 3) * 4);
                __hip_atomic_store(f, call, __ATOMIC_RELAXED, __HIP_MEMORY_SCOPE_SYSTEM);
            }
            if (lane == 0) __hip_atomic_store(cnt, call, __ATOMIC_RELAXED, __HIP_MEMORY_SCOPE_AGENT);
        }
    } else {
        GAS unsigned* cnt = (GAS unsigned*)(wsl + CNTQ_OFF) + b * 32;
        GAS unsigned* tmo = (GAS unsigned*)(wsl + TMO_OFF);
        const unsigned cv = __hip_atomic_load(cnt, __ATOMIC_RELAXED, __HIP_MEMORY_SCOPE_AGENT);
        if (lane < 12) ((volatile LAS unsigned*)(L + CTRL))[lane] = lane == 2 ? 3u : 0u;
        asm volatile("s_waitcnt lgkmcnt(0)" ::: "memory");
        __builtin_amdgcn_s_barrier();
        CFENCE();
        const unsigned call = __builtin_amdgcn_readfirstlane(cv) + 1;
        if (lane == 0) __hip_atomic_store(cnt, call, __ATOMIC_RELAXED, __HIP_MEMORY_SCOPE_AGENT);
        volatile LAS unsigned* cw = (volatile LAS unsigned*)(L + CTRL);
        cw[3] = (call & 1) * XSLOT;
        const unsigned char* fp = wsl + FLAG_OFF + (b & (REP - 1)) * 1024 + mh * 512 + (lane & 31) * 16;
        const unsigned long long t0 = __builtin_amdgcn_s_memrealtime();
        int ready = 0;
        for (;;) {
            v4u f;
            asm volatile("global_load_dwordx4 %0, %1, off sc0 sc1\n\ts_waitcnt vmcnt(0)" : "=&v"(f) : "v"(fp) : "memory");
            const bool ok = min(min(f.x, f.y), min(f.z, f.w)) >= call;
            const unsigned mask = (unsigned)__builtin_amdgcn_ballot_w64(ok) | (0xffffffffu << NREM);
            const int r = mask == 0xffffffffu ? NREM : __builtin_ctz(~mask);
            if (r > ready) {
                ready = r;
                cw[2] = r == NREM ? BIG : (unsigned)(3 + r);
                if (r == NREM) break;
            }
            if (__hip_atomic_load(tmo, __ATOMIC_RELAXED, __HIP_MEMORY_SCOPE_AGENT) == call
                || __builtin_amdgcn_s_memrealtime() - t0 > DEADLINE_TICKS) {
                __hip_atomic_store(tmo, call, __ATOMIC_RELAXED, __HIP_MEMORY_SCOPE_AGENT);
                cw[8] = 1u;
                cw[2] = BIG;
                break;
            }
            __builtin_amdgcn_s_sleep(2);
        }
    }
}

extern "C" void kernel_launch(int rank, int nranks, ncclComm_t comm,
                              hipStream_t stream,
                              void* const* d_in, const int* in_sizes, int n_in,
                              void* d_out, int out_size,
                              const int* in_dtypes, int out_dtype,
                              void* const* d_out_all,
                              void* const* d_ws_all, size_t ws_bytes) {
    if (nranks != NRANK || (unsigned)rank >= (unsigned)NRANK || n_in < 4 || ws_bytes < WS_NEED
        || out_size != ML * NN || in_sizes[0] != 4096 * KSL || in_sizes[1] != 4096 * NN) {
        fprintf(stderr, "unexpected problem shape\n");
        return;
    }
    static bool init = false;
    if (!init) {
        HIP_CHECK(hipFuncSetAttribute((const void*)a2a_gemm, hipFuncAttributeMaxDynamicSharedMemorySize, LDS_BYTES));
        init = true;
    }
    Args a;
    a.x = (const unsigned char*)d_in[0];
    a.w = (const unsigned char*)d_in[1];
    a.sx = (const float*)d_in[2];
    a.sw = (const float*)d_in[3];
    a.out = (float*)d_out;
    for (int p = 0; p < NRANK; ++p) a.ws[p] = (unsigned char*)d_ws_all[p];
    a.rank = rank;
    a.pad = 0;
    hipLaunchKernelGGL(a2a_gemm, dim3(256), dim3(512), LDS_BYTES, stream, a);
    HIP_CHECK(hipGetLastError());
}
```

```cpp
#include <hip/hip_runtime.h>
#include <rccl/rccl.h>
#include <cstdio>

#define HIP_CHECK(expr) do { hipError_t _e = (expr); if (_e != hipSuccess) { \
    fprintf(stderr, "HIP error %s at %s:%d\n", hipGetErrorString(_e), __FILE__, __LINE__); \
    return; } } while (0)

typedef int v2i __attribute__((ext_vector_type(2)));
typedef int v4i __attribute__((ext_vector_type(4)));
typedef unsigned v4u __attribute__((ext_vector_type(4)));
typedef int v8i __attribute__((ext_vector_type(8)));
typedef float v16f __attribute__((ext_vector_type(16)));
#define LAS __attribute__((address_space(3)))
#define GAS __attribute__((address_space(1)))

constexpr int NRANK = 8;
constexpr int ML = 512;
constexpr int NN = 16384;
constexpr int KSL = 512;
constexpr int NSTEP = 32;
constexpr int DEPTH = 8;
constexpr int STAGE = 16384;
constexpr int CTRL = DEPTH * STAGE;
constexpr int LDS_BYTES = CTRL + 64;
constexpr unsigned BIG = 64;

#define XPIECE 2048
constexpr unsigned XBLK = 8 * XPIECE;
constexpr unsigned XSTEP = 8 * XBLK;
constexpr unsigned XSLOT = (NSTEP + 2) * XSTEP;
constexpr unsigned FLAG_OFF = 2 * XSLOT;
constexpr int REP = 8;
constexpr unsigned CNTP_OFF = FLAG_OFF + 65536;
constexpr unsigned CNTQ_OFF = CNTP_OFF + 32768;
constexpr unsigned TMO_OFF = CNTQ_OFF + 32768;
constexpr unsigned DBG_OFF = TMO_OFF + 4096;
constexpr unsigned WS_NEED = DBG_OFF + (1u << 20);
constexpr unsigned long long DEADLINE_TICKS = 50000000ull;
#define XAUX 17
#define WPOL ""
#define SWAP 1
#define POLL_SLEEP 2
#define INFL0 2
#define RAMP 1
#define PF_FIRST 6
#define INFL 2
#define PACE0 90
#define PACE_ROUND 150
#define PACE_BLK 15

struct Args {
    const unsigned char* x;
    const unsigned char* w;
    const float* sx;
    const float* sw;
    float* out;
    unsigned char* ws[NRANK];
    int rank;
    int pad;
};

#define SB() __builtin_amdgcn_sched_barrier(0)
#define CFENCE() asm volatile("" ::: "memory")

#define STAMP(slot) do {} while (0)

__device__ __forceinline__ v8i cat8(v4i lo, v4i hi) {
    return __builtin_shufflevector(lo, hi, 0, 1, 2, 3, 4, 5, 6, 7);
}

__device__ __forceinline__ unsigned xlane_off(int lane) {
    const int g = lane >> 3;
    return (XPIECE == 2048 ? g + (g >= 3) : g) * 128 + (lane & 7) * 16;
}

__device__ __forceinline__ unsigned krow(int st, int rank) {
    if (st < 4) return rank * KSL + st * 128;
    const int t = st - 4, c = t / 7, i = t - c * 7;
    return ((rank + 1 + i) & 7) * KSL + c * 128;
}

__global__ void __launch_bounds__(512, 2) a2a_gemm(Args a) {
    extern __shared__ __attribute__((aligned(16))) unsigned char smem[];
    LAS unsigned char* L = (LAS unsigned char*)smem;
    const int tid = threadIdx.x;
    const int lane = tid & 63;
    const int wave = __builtin_amdgcn_readfirstlane(tid >> 6);
    const int b = blockIdx.x;
    const int rank = a.rank;
    const int idx = b >> 3;
    const int nt = (b & 7) * 16 + (idx >> 1);
    const int mh = idx & 1;
    const int n0 = nt * 128;
    unsigned char* wsl = a.ws[rank];
    STAMP(0);

    if (wave < 4) {
        __builtin_amdgcn_s_setprio(2);
        const int mb = mh * 4 + wave;
        v8i xf[4][2];
        v8i wf[4];
        v16f acc[2][4];
#pragma unroll
        for (int i = 0; i < 2; ++i)
#pragma unroll
            for (int j = 0; j < 4; ++j)
#pragma unroll
                for (int r = 0; r < 16; ++r) acc[i][j][r] = 0.f;
        const __amdgpu_buffer_rsrc_t rx = __builtin_amdgcn_make_buffer_rsrc((void*)wsl, 0, 2 * XSLOT, 0x00020000);
        const unsigned voff = xlane_off(lane);
        __builtin_amdgcn_s_barrier();
        CFENCE();
        unsigned wadr[4];
        {
            const int qq = (lane & 15) >> 1, p = lane & 1, cg = (lane >> 4) & 1, h = lane >> 5;
#pragma unroll
            for (int j = 0; j < 4; ++j)
                wadr[j] = (16 * h + qq) * 128 + ((j ^ ((qq >> 1) & 3)) << 5) + 16 * cg + 8 * p;
        }
        volatile LAS v4u* ctrl = (volatile LAS v4u*)(L + CTRL);
        volatile LAS unsigned* done = (volatile LAS unsigned*)(L + CTRL + 16) + wave;
        unsigned xs;
        {
            v4u c = *ctrl;
            while (__builtin_amdgcn_readfirstlane(c.z) < 1u) { __builtin_amdgcn_s_sleep(1); c = *ctrl; }
            CFENCE();
            xs = __builtin_amdgcn_readfirstlane(c.w) + mb * XBLK;
            STAMP(4);
#pragma unroll
            for (int u = 0; u < 3; ++u)
#pragma unroll
                for (int i = 0; i < 2; ++i) {
                    const unsigned o = xs + (u >> 1) * XSTEP + ((u & 1) * 4 + i * 2) * XPIECE;
                    v4i lo = __builtin_amdgcn_raw_buffer_load_b128(rx, voff, o, XAUX);
                    v4i hi = __builtin_amdgcn_raw_buffer_load_b128(rx, voff, o + XPIECE, XAUX);
                    xf[u][i] = cat8(lo, hi);
                }
            xs += XSTEP + 4 * XPIECE;
            while (__builtin_amdgcn_readfirstlane(c.x) < 1u) { __builtin_amdgcn_s_sleep(1); c = *ctrl; }
        }
        CFENCE();
        STAMP(1);
        auto wread = [&](int j, unsigned stage_off, int kk) -> v8i {
            LAS unsigned char* p = L + stage_off + wadr[j] + kk * 8192;
            v2i r0 = __builtin_amdgcn_ds_read_tr8_b64_v2i32((LAS v2i*)(p));
            v2i r1 = __builtin_amdgcn_ds_read_tr8_b64_v2i32((LAS v2i*)(p + 1024));
            v2i r2 = __builtin_amdgcn_ds_read_tr8_b64_v2i32((LAS v2i*)(p + 4096));
            v2i r3 = __builtin_amdgcn_ds_read_tr8_b64_v2i32((LAS v2i*)(p + 5120));
            return cat8(__builtin_shufflevector(r0, r1, 0, 1, 2, 3), __builtin_shufflevector(r2, r3, 0, 1, 2, 3));
        };
        wf[0] = wread(0, 0, 0);
        wf[1] = wread(1, 0, 0);
        wf[2] = wread(2, 0, 0);
        wf[3] = wf[2];
        xf[3][0] = xf[2][0]; xf[3][1] = xf[2][1];
        SB();
        for (int it = 0; it < NSTEP / 2; ++it) {
            const unsigned so0 = ((2 * it) & (DEPTH - 1)) * STAGE;
            const unsigned so2 = ((2 * it + 2) & (DEPTH - 1)) * STAGE;
            v4u cw;
#pragma unroll
            for (int u = 0; u < 4; ++u) {
                v4i xp[4];
#pragma unroll
                for (int j = 0; j < 4; ++j) {
                    const int s3 = 4 * u + j + 3;
                    const int u3 = s3 >> 2, j3 = s3 & 3;
                    if ((u & 1) == 1 && j == 0) {
                        const unsigned T = 2 * it + (u >> 1);
                        unsigned m = __builtin_amdgcn_readfirstlane(min(cw.x, cw.z));
                        if (__builtin_expect(m <= T + 1, 0)) {
                            while (m <= T + 1) {
                                __builtin_amdgcn_s_sleep(1);
                                cw = *ctrl;
                                m = __builtin_amdgcn_readfirstlane(min(cw.x, cw.z));
                            }
                        }
                        CFENCE();
                    }
                    acc[0][j] = __builtin_amdgcn_mfma_scale_f32_32x32x64_f8f6f4(xf[u][0], wf[j], acc[0][j], 0, 0, 0, 0, 0, 0);
                    acc[1][j] = __builtin_amdgcn_mfma_scale_f32_32x32x64_f8f6f4(xf[u][1], wf[j], acc[1][j], 0, 0, 0, 0, 0, 0);
                    wf[j3] = wread(j3, u3 < 4 ? so0 + (u3 >> 1) * STAGE : so2, u3 & 1);
                    xp[j] = __builtin_amdgcn_raw_buffer_load_b128(rx, voff, xs + j * XPIECE, XAUX);
                    if (j == 1) xf[(u + 3) & 3][0] = cat8(xp[0], xp[1]);
                    if (j == 3) xf[(u + 3) & 3][1] = cat8(xp[2], xp[3]);
                    if (j == 3) xs += (u & 1) ? 4 * XPIECE : XSTEP - 4 * XPIECE;
                    __builtin_amdgcn_sched_group_barrier(0x008, 1, 0);
                    __builtin_amdgcn_sched_group_barrier(0x100, 2, 0);
                    __builtin_amdgcn_sched_group_barrier(0x020, 1, 0);
                    __builtin_amdgcn_sched_group_barrier(0x008, 1, 0);
                    __builtin_amdgcn_sched_group_barrier(0x100, 2, 0);
                    SB();
                    if ((u & 1) == 0 && j == 1) { cw = *ctrl; SB(); }
                    if ((u & 1) == 1 && j == 0) {
                        CFENCE();
                        *done = 2 * it + (u >> 1) + 1;
                        CFENCE();
                        SB();
                    }
                }
            }
        }
        STAMP(2);
        const unsigned bad = *(volatile LAS unsigned*)(L + CTRL + 32);
        const float scale = a.sx[0] * a.sw[0];
        const float sc = bad ? __builtin_nanf("") : scale;
        float* o = a.out + (size_t)(mb * 64 + 4 * (lane >> 5)) * NN + n0 + (lane & 31);
#pragma unroll
        for (int i = 0; i < 2; ++i)
#pragma unroll
            for (int r = 0; r < 16; ++r)
#pragma unroll
                for (int j = 0; j < 4; ++j)
                    o[(size_t)(32 * i + (r & 3) + 8 * (r >> 2)) * NN + 32 * j] = acc[i][j][r] * sc;
        STAMP(3);
    } else if (wave == 4) {
        const int rr = lane >> 3, p = lane & 7;
        const unsigned voff = rr * NN + ((((p >> 1) ^ ((rr >> 1) & 3)) << 5) | ((p & 1) << 4));
        v4i rs;
        {
            const unsigned long long wb = (unsigned long long)(a.w + n0);
            rs.x = __builtin_amdgcn_readfirstlane((int)(unsigned)wb);
            rs.y = __builtin_amdgcn_readfirstlane((int)(unsigned)(wb >> 32) & 0xffff);
            rs.z = 4096 * NN - n0;
            rs.w = 0x00020000;
        }
        const unsigned lbase = (unsigned)(size_t)L;
        auto issue = [&](int s) {
            const unsigned so = krow(s ^ (mh * SWAP), rank) * NN;
            const unsigned m = lbase + (s & (DEPTH - 1)) * STAGE;
#pragma unroll
            for (int q = 0; q < 16; ++q)
                asm volatile("s_mov_b32 m0, %0\n\ts_nop 0\n\tbuffer_load_dwordx4 %1, %2, %3 offen " WPOL " lds"
                             :: "s"(m + q * 1024), "v"(voff), "s"(rs), "s"(so + q * 8 * NN) : "memory");
        };
        asm volatile("s_nop 4" ::: "memory");
        __builtin_amdgcn_s_barrier();
        CFENCE();
        volatile LAS unsigned* full = (volatile LAS unsigned*)(L + CTRL);
        volatile LAS v4u* done = (volatile LAS v4u*)(L + CTRL + 16);
        int issued = 0, pub = 0;
        while (pub < NSTEP) {
            if (issued < NSTEP && issued - pub < (pub < RAMP ? INFL0 : INFL)) {
                const v4u d = *done;
                const int fr = __builtin_amdgcn_readfirstlane(min(min(d.x, d.y), min(d.z, d.w))) + DEPTH;
                CFENCE();
                if (issued < fr) { issue(issued); ++issued; continue; }
            }
            const int n = issued - pub;
            if (n == 0) { __builtin_amdgcn_s_sleep(2); continue; }
            switch (n) {
                case 1: asm volatile("s_waitcnt vmcnt(0)" ::: "memory"); break;
                case 2: asm volatile("s_waitcnt vmcnt(16)" ::: "memory"); break;
                default: asm volatile("s_waitcnt vmcnt(32)" ::: "memory"); break;
            }
            ++pub;
            *full = pub < NSTEP ? (unsigned)pub : BIG;
        }
        STAMP(1);
    } else if (wave == 5) {
        __builtin_amdgcn_s_barrier();
        const unsigned char* wx = a.w + (b & 7) * 2048 + (size_t)(idx * 4) * NN;
        const unsigned l0 = (unsigned)((unsigned long long)wx >> 7);
        const unsigned o0 = ((3 - l0) & 7) * 128, o1 = o0 + 1024;
        const unsigned e = (63 - l0) & 63;
        const unsigned o2 = e < 16 ? e * 128 : o1;
        unsigned dmy = 0;
        for (int s = PF_FIRST; s < NSTEP; ++s) {
            const unsigned char* p = wx + (size_t)krow(s, rank) * NN;
            const unsigned char* p0 = p + o0;
            const unsigned char* p1 = p + o1;
#pragma unroll
            for (int q = 0; q < 8; ++q) {
                asm volatile("s_load_dword %0, %1, %2" : "+s"(dmy) : "s"(p0), "i"((q >> 1) * NN + (q & 1) * 64) : "memory");
                asm volatile("s_load_dword %0, %1, %2" : "+s"(dmy) : "s"(p1), "i"((q >> 1) * NN + (q & 1) * 64) : "memory");
            }
            if (e < 16) {
                const unsigned char* p2 = p + o2;
#pragma unroll
                for (int q = 0; q < 8; ++q)
                    asm volatile("s_load_dword %0, %1, %2" : "+s"(dmy) : "s"(p2), "i"((q >> 1) * NN + (q & 1) * 64) : "memory");
            }
        }
        asm volatile("s_waitcnt lgkmcnt(0)" : "+s"(dmy) :: "memory");
        STAMP(1);
    } else if (wave == 6) {
        __builtin_amdgcn_s_barrier();
    } else {
        const unsigned long long t0 = __builtin_amdgcn_s_memrealtime();
        GAS unsigned* cnt = (GAS unsigned*)(wsl + CNTQ_OFF) + b * 32;
        GAS unsigned* tmo = (GAS unsigned*)(wsl + TMO_OFF);
        const unsigned d = (unsigned)(rank - (b >> 5)) & (NRANK - 1);
        const int pmb = b & 7, c = (b >> 3) & 3;
        unsigned cv = __hip_atomic_load(cnt, __ATOMIC_RELAXED, __HIP_MEMORY_SCOPE_AGENT);
        v4u v[8];
        {
            const GAS unsigned char* src = (const GAS unsigned char*)a.x + (size_t)(d * ML + pmb * 64 + (lane & 31)) * KSL
                                           + c * 128 + (lane >> 5) * 16;
#pragma unroll
            for (int q = 0; q < 8; ++q)
                v[q] = *(const GAS v4u*)(src + ((q >> 1) & 1) * 32 * KSL + (q >> 2) * 64 + (q & 1) * 32);
        }
        if (lane < 12) ((volatile LAS unsigned*)(L + CTRL))[lane] = 0u;
        asm volatile("s_waitcnt lgkmcnt(0)" ::: "memory");
        __builtin_amdgcn_s_barrier();
        CFENCE();
        const bool self = b < 32;
        const int step = self ? c : 3 + 7 * c + (b >> 5);
        const int t = step ^ ((pmb >> 2) * SWAP);
        const int ord = (pmb & 3) + (t > (step ^ (((pmb >> 2) ^ 1) * SWAP)) ? 4 : 0);
        const unsigned long long when = self ? 0 : PACE0 + c * PACE_ROUND + ord * PACE_BLK;
        unsigned char* pw = a.ws[d];
        asm volatile("s_waitcnt vmcnt(0)" : "+v"(v[0]), "+v"(v[1]), "+v"(v[2]), "+v"(v[3]),
                                            "+v"(v[4]), "+v"(v[5]), "+v"(v[6]), "+v"(v[7]), "+v"(cv) :: "memory");
        const unsigned call = __builtin_amdgcn_readfirstlane(cv) + 1;
        volatile LAS unsigned* cw = (volatile LAS unsigned*)(L + CTRL);
        cw[3] = (call & 1) * XSLOT;
        unsigned char* dst = pw + (call & 1) * XSLOT + t * XSTEP + pmb * XBLK + xlane_off(lane);
        const unsigned char* fp = wsl + FLAG_OFF + (b & (REP - 1)) * 1024 + mh * 512 + (lane & 31) * 16;
        int ready = 0;
        bool sent = false;
        for (unsigned spin = 1;; ++spin) {
            if (!sent) {
                const unsigned long long el = __builtin_amdgcn_s_memrealtime() - t0;
                if (el >= when) {
#pragma unroll
                    for (int q = 0; q < 8; ++q)
                        asm volatile("global_store_dwordx4 %0, %1, off sc0 sc1\n\ts_nop 1" :: "v"(dst + q * XPIECE), "v"(v[q]) : "memory");
                    asm volatile("s_waitcnt vmcnt(0)" ::: "memory");
                    if (lane < REP) {
                        GAS unsigned* f = (GAS unsigned*)(pw + FLAG_OFF + lane * 1024 + (pmb >> 2) * 512 + t * 16 + (pmb & 3) * 4);
                        __hip_atomic_store(f, call, __ATOMIC_RELAXED, __HIP_MEMORY_SCOPE_SYSTEM);
                    }
                    sent = true;
                } else if (ready == NSTEP || when - el < 40) {
                    __builtin_amdgcn_s_sleep(1);
                    continue;
                }
            }
            if (ready == NSTEP) break;
            v4u f;
            asm volatile("global_load_dwordx4 %0, %1, off sc0 sc1\n\ts_waitcnt vmcnt(0)" : "=&v"(f) : "v"(fp) : "memory");
            const bool ok = min(min(f.x, f.y), min(f.z, f.w)) >= call;
            const unsigned mask = (unsigned)__builtin_amdgcn_ballot_w64(ok);
            const int r = mask == 0xffffffffu ? NSTEP : __builtin_ctz(~mask);
            if (r > ready) {
                cw[2] = r == NSTEP ? BIG : (unsigned)(r - 1);
                ready = r;
                continue;
            }
            if ((spin & 63) == 0
                && (__hip_atomic_load(tmo, __ATOMIC_RELAXED, __HIP_MEMORY_SCOPE_AGENT) == call
                    || __builtin_amdgcn_s_memrealtime() - t0 > DEADLINE_TICKS)) {
                __hip_atomic_store(tmo, call, __ATOMIC_RELAXED, __HIP_MEMORY_SCOPE_AGENT);
                cw[8] = 1u;
                cw[2] = BIG;
                ready = NSTEP;
                continue;
            }
            __builtin_amdgcn_s_sleep(POLL_SLEEP);
        }
        if (lane == 0) __hip_atomic_store(cnt, call, __ATOMIC_RELAXED, __HIP_MEMORY_SCOPE_AGENT);
    }
}

extern "C" void kernel_launch(int rank, int nranks, ncclComm_t comm,
                              hipStream_t stream,
                              void* const* d_in, const int* in_sizes, int n_in,
                              void* d_out, int out_size,
                              const int* in_dtypes, int out_dtype,
                              void* const* d_out_all,
                              void* const* d_ws_all, size_t ws_bytes) {
    if (nranks != NRANK || (unsigned)rank >= (unsigned)NRANK || n_in < 4 || ws_bytes < WS_NEED
        || out_size != ML * NN || in_sizes[0] != 4096 * KSL || in_sizes[1] != 4096 * NN) {
        fprintf(stderr, "unexpected problem shape\n");
        return;
    }
    static bool init = false;
    if (!init) {
        HIP_CHECK(hipFuncSetAttribute((const void*)a2a_gemm, hipFuncAttributeMaxDynamicSharedMemorySize, LDS_BYTES));
        init = true;
    }
    Args a;
    a.x = (const unsigned char*)d_in[0];
    a.w = (const unsigned char*)d_in[1];
    a.sx = (const float*)d_in[2];
    a.sw = (const float*)d_in[3];
    a.out = (float*)d_out;
    for (int p = 0; p < NRANK; ++p) a.ws[p] = (unsigned char*)d_ws_all[p];
    a.rank = rank;
    a.pad = 0;
    hipLaunchKernelGGL(a2a_gemm, dim3(256), dim3(512), LDS_BYTES, stream, a);
    HIP_CHECK(hipGetLastError());
}
```

```cpp
#include <hip/hip_runtime.h>
#include <rccl/rccl.h>
#include <cstdio>
#include <type_traits>

#define HIP_CHECK(expr) do { hipError_t _e = (expr); if (_e != hipSuccess) { \
    fprintf(stderr, "HIP error %s at %s:%d\n", hipGetErrorString(_e), __FILE__, __LINE__); \
    return; } } while (0)

typedef int v2i __attribute__((ext_vector_type(2)));
typedef int v4i __attribute__((ext_vector_type(4)));
typedef unsigned v4u __attribute__((ext_vector_type(4)));
typedef int v8i __attribute__((ext_vector_type(8)));
typedef unsigned v16u __attribute__((ext_vector_type(16)));
typedef float v16f __attribute__((ext_vector_type(16)));
typedef unsigned long long u64;
#define LAS __attribute__((address_space(3)))
#define GAS __attribute__((address_space(1)))

constexpr int NRANK = 8;
constexpr int ML = 512;
constexpr int NN = 16384;
constexpr int KSL = 512;
constexpr int NSTEP = 32;
constexpr int NLOC = 4;
constexpr int DEPTH = 8;
constexpr int STAGE = 16384;
constexpr int HELD = 3;
constexpr int PUSHB = (DEPTH + 1) * STAGE;
constexpr int CTRL = PUSHB + 8192;
constexpr int LDS_BYTES = CTRL + 64;
constexpr unsigned BIG = 64;

constexpr unsigned XPIECE = 2048;
constexpr unsigned XBLK = 8 * XPIECE;
constexpr unsigned XSTEP = 8 * XBLK;
constexpr unsigned XSLOT = (NSTEP - NLOC) * XSTEP;
constexpr unsigned FLAG_OFF = 2 * XSLOT;
constexpr unsigned CNT_OFF = FLAG_OFF + 65536;
constexpr unsigned TMO_OFF = CNT_OFF + 32768;
constexpr unsigned DBG_OFF = TMO_OFF + 4096;
constexpr unsigned WS_NEED = DBG_OFF + (1u << 20);
constexpr u64 DEADLINE_TICKS = 50000000ull;
#define REP 8
#define OAUX 16
#define LPRIO 3
#define CPRIO 3
#define SWAP 1
#define POLL_SLEEP 1
#define PF_FIRST 5
#define INFL0 1
#define INFL 2
#define PACE0 70
#define PACE1 420
#define LDELAY0 2
#define LDELAY1 14
#define XDELAY 8
#define PACE_ROUND 150
#define PACE_BLK 15

struct Args {
    const unsigned char* x;
    const unsigned char* w;
    const float* sx;
    const float* sw;
    float* out;
    int rank;
    int pad;
    unsigned char* ws[NRANK];
};

#define SB() __builtin_amdgcn_sched_barrier(0)
#define CFENCE() asm volatile("" ::: "memory")
#define NOW() __builtin_amdgcn_s_memrealtime()

#define TS(v) do {} while (0)

__device__ __forceinline__ v8i cat8(v4i lo, v4i hi) {
    return __builtin_shufflevector(lo, hi, 0, 1, 2, 3, 4, 5, 6, 7);
}

__device__ __forceinline__ v4i make_rsrc(const void* p, unsigned bytes) {
    const u64 q = (u64)p;
    v4i r;
    r.x = __builtin_amdgcn_readfirstlane((int)(unsigned)q);
    r.y = __builtin_amdgcn_readfirstlane((int)(unsigned)(q >> 32) & 0xffff);
    r.z = bytes;
    r.w = 0x00020000;
    return r;
}

__device__ __forceinline__ void dma16(unsigned m, unsigned voff, v4i rsrc, unsigned soff) {
    asm volatile("s_mov_b32 m0, %0\n\ts_nop 0\n\tbuffer_load_dwordx4 %1, %2, %3 offen lds"
                 :: "s"(m), "v"(voff), "s"(rsrc), "s"(soff) : "memory");
}

#define WPOL ""
__device__ __forceinline__ void dma16w(unsigned m, unsigned voff, v4i rsrc, unsigned soff) {
    asm volatile("s_mov_b32 m0, %0\n\ts_nop 0\n\tbuffer_load_dwordx4 %1, %2, %3 offen " WPOL " lds"
                 :: "s"(m), "v"(voff), "s"(rsrc), "s"(soff) : "memory");
}

__device__ __forceinline__ unsigned xlane_off(int lane) {
    const int g = lane >> 3;
    return (g + (g >= 3)) * 128 + (lane & 7) * 16;
}

__device__ __forceinline__ unsigned krow(int st, int rank) {
    if (st < NLOC) return rank * KSL + st * 128;
    const int t = st - NLOC, c = (t * 37) >> 8, i = t - c * 7;
    return ((rank + 1 + i) & 7) * KSL + c * 128;
}

template <int V> using IC = std::integral_constant<int, V>;

__global__ void __launch_bounds__(512, 2) a2a_gemm(Args a) {
    extern __shared__ __attribute__((aligned(16))) unsigned char smem[];
    LAS unsigned char* L = (LAS unsigned char*)smem;
    const unsigned L0 = (unsigned)(size_t)L;
    const int tid = threadIdx.x;
    const int lane = tid & 63;
    const int wave = __builtin_amdgcn_readfirstlane(tid >> 6);
    const int b = blockIdx.x;
    const int rank = a.rank;
    const int idx = b >> 3;
    const int nt = (b & 7) * 16 + (idx >> 1);
    const int mh = idx & 1;
    const int n0 = nt * 128;
    unsigned char* wsl = a.ws[0];
    const unsigned lv0 = (lane >> 3) * KSL + (((lane & 7) ^ (lane >> 4)) << 4);
    const unsigned lv1 = lv0 ^ 64;
    const unsigned xrd = (lane & 31) * 128 + (((lane >> 5) ^ ((lane >> 1) & 7)) << 4);
    TS(t_start);
    if (tid < 12) ((volatile LAS unsigned*)(L + CTRL))[tid] = tid == 2 ? NLOC - 1 : 0u;
    asm volatile("s_waitcnt lgkmcnt(0)\n\ts_barrier" ::: "memory");
    TS(t_bar);

    if (wave == 7) {
        __builtin_amdgcn_s_setprio(CPRIO);
        const u64 t0 = NOW();
        const unsigned dj = (unsigned)(b >> 5) & (NRANK - 1);
        const int pmb = b & 7, c = (b >> 3) & 3;
        const bool self = dj == 0;
        GAS unsigned* cnt = (GAS unsigned*)(wsl + CNT_OFF) + b * 32;
        GAS unsigned* tmo = (GAS unsigned*)(wsl + TMO_OFF);
        const v4i rp = make_rsrc(a.x + (size_t)(((rank - dj) & (NRANK - 1)) * ML + pmb * 64) * KSL, 64 * KSL);
        auto fetch = [&]() {
#pragma unroll
            for (int q = 0; q < 8; ++q) dma16(L0 + PUSHB + q * 1024, (q & 1) ? lv1 : lv0, rp, q * 8 * KSL + c * 128);
        };
        asm volatile("s_nop 4" ::: "memory");
        int state = self ? 2 : 0;
        if (!self && c == 0) { fetch(); state = 1; }
        unsigned cv;
        asm volatile("s_load_dword %0, %1, 0 glc\n\ts_waitcnt lgkmcnt(0)" : "=&s"(cv) : "s"(cnt) : "memory");
        const unsigned call = cv + 1;
        volatile LAS unsigned* cw = (volatile LAS unsigned*)(L + CTRL);
        cw[3] = (call & 1) * XSLOT + 1;
        const int step = NLOC - 1 + 7 * c + (int)dj;
        const int t = step ^ ((pmb >> 2) * SWAP);
        const int ord = (pmb & 3) + (t > (step ^ (((pmb >> 2) ^ 1) * SWAP)) ? 4 : 0);
        const u64 when = (c == 0 ? PACE0 : PACE1 + (c - 1) * PACE_ROUND) + ord * PACE_BLK;
        unsigned char* pw = a.ws[dj];
        unsigned char* dst = pw + (call & 1) * XSLOT + (self ? 0 : t - NLOC) * XSTEP + pmb * XBLK + xlane_off(lane);
        const unsigned char* fb = wsl + FLAG_OFF + (b & (REP - 1)) * 1024 + mh * 512;
        const unsigned char* fp = fb + (lane & 31) * 16;
        int ready = NLOC;
        for (unsigned spin = 1;; ++spin) {
            if (state < 2) {
                const u64 el = NOW() - t0;
                if (state == 0 && el + 120 >= when) { fetch(); state = 1; }
                if (state == 1 && el >= when) {
                    asm volatile("s_waitcnt vmcnt(0)" ::: "memory");
                    v4u v[8];
#pragma unroll
                    for (int q = 0; q < 8; ++q)
                        v[q] = *(LAS v4u*)(L + PUSHB + (xrd ^ (((q >> 2) * 4 + (q & 1) * 2) << 4)) + ((q >> 1) & 1) * 4096);
#pragma unroll
                    for (int q = 0; q < 8; ++q)
                        asm volatile("global_store_dwordx4 %0, %1, off sc0 sc1\n\ts_nop 1" :: "v"(dst + q * XPIECE), "v"(v[q]) : "memory");
                    asm volatile("s_waitcnt vmcnt(0)" ::: "memory");
                    if (lane < REP) {
                        GAS unsigned* f = (GAS unsigned*)(pw + FLAG_OFF + lane * 1024 + (pmb >> 2) * 512 + t * 16 + (pmb & 3) * 4);
                        __hip_atomic_store(f, call, __ATOMIC_RELAXED, __HIP_MEMORY_SCOPE_SYSTEM);
                    }
                    state = 2;
                } else if (ready == NSTEP || when - el < 40 || c == 0) {
                    __builtin_amdgcn_s_sleep(1);
                    continue;
                }
            }
            if (ready == NSTEP) break;
            int r;
            if ((spin & 31) != 0) {
                const int w0 = ready & ~3;
                const int w1 = w0 + 4 < NSTEP ? w0 + 4 : w0;
                v16u g, h;
                asm volatile("s_load_dwordx16 %0, %2, 0 glc\n\ts_load_dwordx16 %1, %3, 0 glc\n\ts_waitcnt lgkmcnt(0)"
                             : "=&s"(g), "=&s"(h) : "s"(fb + w0 * 16), "s"(fb + w1 * 16) : "memory");
                unsigned m = 0;
#pragma unroll
                for (int q = 0; q < 4; ++q) {
                    m |= (min(min(g[4 * q], g[4 * q + 1]), min(g[4 * q + 2], g[4 * q + 3])) >= call ? 1u : 0u) << q;
                    m |= (min(min(h[4 * q], h[4 * q + 1]), min(h[4 * q + 2], h[4 * q + 3])) >= call ? 16u : 0u) << q;
                }
                r = min(w0 + __builtin_ctz(~m), NSTEP);
            } else {
                v4u f;
                asm volatile("global_load_dwordx4 %0, %1, off sc0 sc1\n\ts_waitcnt vmcnt(0)" : "=&v"(f) : "v"(fp) : "memory");
                const bool ok = min(min(f.x, f.y), min(f.z, f.w)) >= call;
                const unsigned mask = (unsigned)__builtin_amdgcn_ballot_w64(ok) | ((1u << NLOC) - 1);
                r = mask == 0xffffffffu ? NSTEP : __builtin_ctz(~mask);
            }
            if (r > ready) {
                cw[2] = r == NSTEP ? BIG : (unsigned)(r - 1);
                ready = r;
                continue;
            }
            if ((spin & 63) == 0
                && (__hip_atomic_load(tmo, __ATOMIC_RELAXED, __HIP_MEMORY_SCOPE_AGENT) == call
                    || NOW() - t0 > DEADLINE_TICKS)) {
                __hip_atomic_store(tmo, call, __ATOMIC_RELAXED, __HIP_MEMORY_SCOPE_AGENT);
                cw[8] = 1u;
                cw[2] = BIG;
                ready = NSTEP;
                continue;
            }
            __builtin_amdgcn_s_sleep(POLL_SLEEP);
        }
        if (lane == 0) __hip_atomic_store(cnt, call, __ATOMIC_RELAXED, __HIP_MEMORY_SCOPE_AGENT);
    } else if (wave < 4) {
        const int mb = mh * 4 + wave;
        const unsigned xreg = (DEPTH - wave) * STAGE;
        const v4i rl = make_rsrc(a.x + (size_t)(rank * ML + mb * 64) * KSL, 64 * KSL);
        auto xstage = [&](int st, int q) {
            dma16(L0 + xreg + (st & 1) * 8192 + q * 1024, (q & 1) ? lv1 : lv0, rl, q * 8 * KSL + (st ^ (mh * SWAP)) * 128);
        };
        __builtin_amdgcn_s_sleep(XDELAY);
#pragma unroll
        for (int q = 0; q < 16; ++q) xstage(q >> 3, q & 7);
        float scale;
        {
            float s0, s1;
            asm volatile("s_load_dword %0, %2, 0\n\ts_load_dword %1, %3, 0\n\ts_waitcnt lgkmcnt(0)"
                         : "=&s"(s0), "=&s"(s1) : "s"(a.sx), "s"(a.sw) : "memory");
            scale = __builtin_bit_cast(float, __builtin_amdgcn_readfirstlane(__builtin_bit_cast(int, s0 * s1)));
        }
        __builtin_amdgcn_s_setprio(2);
        v8i xf[4][2];
        v8i wf[4];
        v16f acc[2][4];
#pragma unroll
        for (int i = 0; i < 2; ++i)
#pragma unroll
            for (int j = 0; j < 4; ++j)
#pragma unroll
                for (int r = 0; r < 16; ++r) acc[i][j][r] = 0.f;
        const __amdgpu_buffer_rsrc_t rx = __builtin_amdgcn_make_buffer_rsrc((void*)wsl, 0, 2 * XSLOT, 0x00020000);
        unsigned voff = xlane_off(lane);
        float* obase = a.out + (size_t)mb * 64 * NN + n0;
        asm volatile("" : "+s"(scale), "+s"(obase));
        const __amdgpu_buffer_rsrc_t ro = __builtin_amdgcn_make_buffer_rsrc((void*)obase, 0, (64 * NN - n0) * 4, 0x00020000);
        unsigned wadr[4];
        {
            const int qq = (lane & 15) >> 1, p = lane & 1, cg = (lane >> 4) & 1, h = lane >> 5;
#pragma unroll
            for (int j = 0; j < 4; ++j)
                wadr[j] = (16 * h + qq) * 128 + ((j ^ ((qq >> 1) & 3)) << 5) + 16 * cg + 8 * p;
        }
        unsigned xadr = xreg + xrd;
        volatile LAS v4u* ctrl = (volatile LAS v4u*)(L + CTRL);
        volatile LAS unsigned* done = (volatile LAS unsigned*)(L + CTRL + 16) + wave;
        asm volatile("" : "+v"(acc[0][0]), "+v"(acc[0][1]), "+v"(acc[0][2]), "+v"(acc[0][3]),
                          "+v"(acc[1][0]), "+v"(acc[1][1]), "+v"(acc[1][2]), "+v"(acc[1][3]),
                          "+v"(wadr[0]), "+v"(wadr[1]), "+v"(wadr[2]), "+v"(wadr[3]), "+v"(voff), "+v"(xadr));
        auto xlds = [&](int u, int i) -> v8i {
            LAS unsigned char* p = L + (u >> 1) * 8192 + i * 4096;
            return cat8(*(LAS v4i*)(p + (xadr ^ ((u & 1) * 64))), *(LAS v4i*)(p + (xadr ^ ((u & 1) * 64 + 32))));
        };
        auto wread = [&](int j, unsigned stage_off, int kk) -> v8i {
            LAS unsigned char* p = L + stage_off + wadr[j] + kk * 8192;
            v2i r0 = __builtin_amdgcn_ds_read_tr8_b64_v2i32((LAS v2i*)(p));
            v2i r1 = __builtin_amdgcn_ds_read_tr8_b64_v2i32((LAS v2i*)(p + 1024));
            v2i r2 = __builtin_amdgcn_ds_read_tr8_b64_v2i32((LAS v2i*)(p + 4096));
            v2i r3 = __builtin_amdgcn_ds_read_tr8_b64_v2i32((LAS v2i*)(p + 5120));
            return cat8(__builtin_shufflevector(r0, r1, 0, 1, 2, 3), __builtin_shufflevector(r2, r3, 0, 1, 2, 3));
        };
        asm volatile("s_waitcnt vmcnt(8)" ::: "memory");
        TS(t_x);
        xf[0][0] = xlds(0, 0); xf[0][1] = xlds(0, 1);
        xf[1][0] = xlds(1, 0); xf[1][1] = xlds(1, 1);
        v4u cw = *ctrl;
        while (__builtin_amdgcn_readfirstlane(cw.x) < 1u) { __builtin_amdgcn_s_sleep(1); cw = *ctrl; }
        CFENCE();
        TS(t_w);
        wf[0] = wread(0, 0, 0);
        wf[1] = wread(1, 0, 0);
        wf[2] = wread(2, 0, 0);
        wf[3] = wf[2];
        xf[2][0] = xf[0][0]; xf[2][1] = xf[0][0]; xf[3][0] = xf[0][0]; xf[3][1] = xf[0][0];
        unsigned xs = 0;
        SB();
        auto sub = [&](auto uc, auto xc, auto lc, unsigned T, unsigned sc_, unsigned sn_) {
            constexpr int u = decltype(uc)::value, XS = decltype(xc)::value;
            constexpr bool LAST = decltype(lc)::value;
            v4i xp[4];
            if constexpr ((u & 1) == 1 && !LAST) {
                unsigned m = __builtin_amdgcn_readfirstlane(min(min(cw.x, cw.y), cw.z));
                if (__builtin_expect(m <= T + 1, 0)) {
                    while (m <= T + 1) {
                        __builtin_amdgcn_s_sleep(1);
                        cw = *ctrl;
                        m = __builtin_amdgcn_readfirstlane(min(min(cw.x, cw.y), cw.z));
                    }
                }
                CFENCE();
            }
            if constexpr (XS == 4 || XS == 5) asm volatile("s_waitcnt vmcnt(8)" ::: "memory");
#pragma unroll
            for (int j = 0; j < 4; ++j) {
                if (XS == 6 && j == 2) asm volatile("s_waitcnt vmcnt(0)" ::: "memory");
                if (XS == 2 && j == 1) {
                    if (__builtin_expect(__builtin_amdgcn_readfirstlane(cw.y) < 1u, 0))
                        do { cw = *ctrl; } while (__builtin_amdgcn_readfirstlane(cw.y) < 1u);
                    CFENCE();
                }
                acc[0][j] = __builtin_amdgcn_mfma_scale_f32_32x32x64_f8f6f4(xf[u][0], wf[j], acc[0][j], 0, 0, 0, 0, 0, 0);
                if constexpr (XS == 2 || XS == 3) {
                    SB();
                    if (XS == 3 && j == 0) asm volatile("s_waitcnt lgkmcnt(0)" ::: "memory");
                    xstage(XS, 2 * j);
                    SB();
                }
                acc[1][j] = __builtin_amdgcn_mfma_scale_f32_32x32x64_f8f6f4(xf[u][1], wf[j], acc[1][j], 0, 0, 0, 0, 0, 0);
                if (j == 0) wf[3] = wread(3, sc_, u & 1);
                else if (!LAST) wf[j - 1] = wread(j - 1, sn_, (u + 1) & 1);
                if constexpr (XS == 1) {
                    xp[j] = __builtin_amdgcn_raw_buffer_load_b128(rx, voff, xs + j * XPIECE, 17);
                    if (j == 1) xf[(u + 3) & 3][0] = cat8(xp[0], xp[1]);
                    if (j == 3) xf[(u + 3) & 3][1] = cat8(xp[2], xp[3]);
                    if (j == 3) xs += (u & 1) ? 4 * XPIECE : XSTEP - 4 * XPIECE;
                }
                if constexpr (XS == 4) xf[2 + (j >> 1)][j & 1] = xlds(2 + (j >> 1), j & 1);
                if constexpr (XS == 5) xf[j >> 1][j & 1] = xlds(j >> 1, j & 1);
                if constexpr (XS == 6) { if (j >= 2) { xf[j][0] = xlds(j, 0); xf[j][1] = xlds(j, 1); } }
                if constexpr (XS == 2 || XS == 3) {
                    __builtin_amdgcn_sched_group_barrier(0x008, 1, 0);
                    __builtin_amdgcn_sched_group_barrier(0x100, 4, 0);
                    SB();
                    xstage(XS, 2 * j + 1);
                } else {
                    __builtin_amdgcn_sched_group_barrier(0x008, 1, 0);
                    __builtin_amdgcn_sched_group_barrier(0x100, XS >= 4 ? 4 : 2, 0);
                    __builtin_amdgcn_sched_group_barrier(0x020, 1, 0);
                    __builtin_amdgcn_sched_group_barrier(0x008, 1, 0);
                    __builtin_amdgcn_sched_group_barrier(0x100, XS >= 4 ? 4 : 2, 0);
                }
                SB();
                if ((u & 1) == 0 && j == 1) { cw = *ctrl; SB(); }
                if (((u & 1) == 1 && j == 0 && XS <= 1) || (XS == 6 && j == 3)) {
                    CFENCE();
                    *done = (XS == 6 ? T : T + 1) + HELD;
                    CFENCE();
                    SB();
                }
            }
        };
        using NO = std::false_type; using YES = std::true_type;
#define HT(i)
        HT(0);
        sub(IC<0>{}, IC<2>{}, NO{}, 0, 0, 0);
        HT(1);
        sub(IC<1>{}, IC<4>{}, NO{}, 0, 0, STAGE);
        HT(2);
        sub(IC<2>{}, IC<3>{}, NO{}, 1, STAGE, STAGE);
        HT(3);
        sub(IC<3>{}, IC<5>{}, NO{}, 1, STAGE, 2 * STAGE);
        HT(4);
        sub(IC<0>{}, IC<6>{}, NO{}, 2, 2 * STAGE, 2 * STAGE);
        HT(5);
        if (__builtin_expect(__builtin_amdgcn_readfirstlane(cw.w) == 0u, 0))
            do { __builtin_amdgcn_s_sleep(1); cw = *ctrl; } while (__builtin_amdgcn_readfirstlane(cw.w) == 0u);
        xs = __builtin_amdgcn_readfirstlane(cw.w) - 1 + mb * XBLK;
        for (unsigned T = 2; T < NSTEP - 2; T += 2) {
            const unsigned so0 = (T & (DEPTH - 1)) * STAGE;
            const unsigned so2 = ((T + 2) & (DEPTH - 1)) * STAGE;
            sub(IC<1>{}, IC<1>{}, NO{}, T, so0, so0 + STAGE);
            sub(IC<2>{}, IC<1>{}, NO{}, T + 1, so0 + STAGE, so0 + STAGE);
            sub(IC<3>{}, IC<1>{}, NO{}, T + 1, so0 + STAGE, so2);
            sub(IC<0>{}, IC<1>{}, NO{}, T + 2, so2, so2);
        }
        sub(IC<1>{}, IC<0>{}, NO{}, NSTEP - 2, 6 * STAGE, 7 * STAGE);
        sub(IC<2>{}, IC<0>{}, NO{}, NSTEP - 1, 7 * STAGE, 7 * STAGE);
        sub(IC<3>{}, IC<0>{}, YES{}, NSTEP - 1, 7 * STAGE, 7 * STAGE);
        TS(t_end);
        const unsigned bad = *(volatile LAS unsigned*)(L + CTRL + 32);
        const float sc = bad ? __builtin_nanf("") : scale;
        const unsigned vo = (4 * (lane >> 5) * NN + (lane & 31)) * 4;
#pragma unroll
        for (int i = 0; i < 2; ++i)
#pragma unroll
            for (int r = 0; r < 16; ++r)
#pragma unroll
                for (int j = 0; j < 4; ++j)
                    __builtin_amdgcn_raw_buffer_store_b32(__builtin_bit_cast(unsigned, acc[i][j][r] * sc), ro, vo + j * 128,
                                                          (32 * i + (r & 3) + 8 * (r >> 2)) * NN * 4, OAUX);
    } else if (wave < 6) {
        __builtin_amdgcn_s_setprio(LPRIO);
        const int ld = wave - 4;
        const int rr = lane >> 3, p = lane & 7;
        const unsigned voff = rr * NN + ((((p >> 1) ^ ((rr >> 1) & 3)) << 5) | ((p & 1) << 4));
        const v4i rs = make_rsrc(a.w + n0, 4096 * NN - n0);
        auto issue = [&](int s) {
            const unsigned so = (krow(s ^ (mh * SWAP), rank) + ld * 64) * NN;
            const unsigned m = L0 + (s & (DEPTH - 1)) * STAGE + ld * 8192;
#pragma unroll
            for (int q = 0; q < 8; ++q) dma16w(m + q * 1024, voff, rs, so + q * 8 * NN);
        };
        if (ld == 0) __builtin_amdgcn_s_sleep(LDELAY0); else __builtin_amdgcn_s_sleep(LDELAY1);
        issue(0);
        TS(t_iss);
        volatile LAS unsigned* full = (volatile LAS unsigned*)(L + CTRL) + ld;
        volatile LAS v4u* done = (volatile LAS v4u*)(L + CTRL + 16);
        int issued = 1, pub = 0;
        while (pub < NSTEP) {
            if (issued < NSTEP && issued - pub < (pub == 0 ? INFL0 : INFL)) {
                const v4u d = *done;
                const int fr = __builtin_amdgcn_readfirstlane(min(min(d.x, d.y), min(d.z, d.w))) + DEPTH - HELD;
                CFENCE();
                if (issued < fr) { issue(issued); ++issued; continue; }
            }
            const int n = issued - pub;
            if (n == 0) { __builtin_amdgcn_s_sleep(2); continue; }
            switch (n) {
                case 1: asm volatile("s_waitcnt vmcnt(0)" ::: "memory"); break;
                case 2: asm volatile("s_waitcnt vmcnt(8)" ::: "memory"); break;
                case 3: asm volatile("s_waitcnt vmcnt(16)" ::: "memory"); break;
                default: asm volatile("s_waitcnt vmcnt(24)" ::: "memory"); break;
            }
            ++pub;
            *full = pub < NSTEP ? (unsigned)pub : BIG;
        }
    } else {
        {
            const unsigned char* px = a.x + (size_t)(rank * ML + mh * 256 + (idx >> 1) * 16) * KSL + lane * 64;
            unsigned t0_, t1_;
            asm volatile("global_load_dword %0, %2, off\n\tglobal_load_dword %1, %3, off\n\ts_waitcnt vmcnt(0)"
                         : "=&v"(t0_), "=&v"(t1_) : "v"(px), "v"(px + 4096) : "memory");
        }
        TS(t_xpf);
#define PF_DELAY 300
        { const u64 t0 = NOW(); while (NOW() - t0 < PF_DELAY) __builtin_amdgcn_s_sleep(8); }
        const unsigned char* wx = a.w + (b & 7) * 2048 + (size_t)(idx * 4) * NN;
        const unsigned l0 = (unsigned)((u64)wx >> 7);
        const unsigned o0 = ((3 - l0) & 7) * 128, o1 = o0 + 1024;
        const unsigned e = (63 - l0) & 63;
        const unsigned o2 = e < 16 ? e * 128 : o1;
        unsigned dmy = 0;
        for (int s = PF_FIRST; s < NSTEP; ++s) {
            const unsigned char* p = wx + (size_t)krow(s, rank) * NN;
            const unsigned char* p0 = p + o0;
            const unsigned char* p1 = p + o1;
#pragma unroll
            for (int q = 0; q < 8; ++q) {
                asm volatile("s_load_dword %0, %1, %2" : "+s"(dmy) : "s"(p0), "i"((q >> 1) * NN + (q & 1) * 64) : "memory");
                asm volatile("s_load_dword %0, %1, %2" : "+s"(dmy) : "s"(p1), "i"((q >> 1) * NN + (q & 1) * 64) : "memory");
            }
            if (e < 16) {
                const unsigned char* p2 = p + o2;
#pragma unroll
                for (int q = 0; q < 8; ++q)
                    asm volatile("s_load_dword %0, %1, %2" : "+s"(dmy) : "s"(p2), "i"((q >> 1) * NN + (q & 1) * 64) : "memory");
            }
        }
        asm volatile("s_waitcnt lgkmcnt(0)" : "+s"(dmy) :: "memory");
    }
}

extern "C" void kernel_launch(int rank, int nranks, ncclComm_t comm,
                              hipStream_t stream,
                              void* const* d_in, const int* in_sizes, int n_in,
                              void* d_out, int out_size,
                              const int* in_dtypes, int out_dtype,
                              void* const* d_out_all,
                              void* const* d_ws_all, size_t ws_bytes) {
    if (nranks != NRANK || (unsigned)rank >= (unsigned)NRANK || n_in < 4 || ws_bytes < WS_NEED
        || out_size != ML * NN || in_sizes[0] != 4096 * KSL || in_sizes[1] != 4096 * NN) {
        fprintf(stderr, "unexpected problem shape\n");
        return;
    }
    static bool init = false;
    if (!init) {
        HIP_CHECK(hipFuncSetAttribute((const void*)a2a_gemm, hipFuncAttributeMaxDynamicSharedMemorySize, LDS_BYTES));
        init = true;
    }
    Args a;
    a.x = (const unsigned char*)d_in[0];
    a.w = (const unsigned char*)d_in[1];
    a.sx = (const float*)d_in[2];
    a.sw = (const float*)d_in[3];
    a.out = (float*)d_out;
    a.rank = rank;
    a.pad = 0;
    for (int j = 0; j < NRANK; ++j) a.ws[j] = (unsigned char*)d_ws_all[(rank - j) & (NRANK - 1)];
    hipLaunchKernelGGL(a2a_gemm, dim3(256), dim3(512), LDS_BYTES, stream, a);
    HIP_CHECK(hipGetLastError());
}
```

```cpp
#include <hip/hip_runtime.h>
#include <rccl/rccl.h>
#include <cstdio>
#include <type_traits>

#define HIP_CHECK(expr) do { hipError_t _e = (expr); if (_e != hipSuccess) { \
    fprintf(stderr, "HIP error %s at %s:%d\n", hipGetErrorString(_e), __FILE__, __LINE__); \
    return; } } while (0)

typedef int v2i __attribute__((ext_vector_type(2)));
typedef int v4i __attribute__((ext_vector_type(4)));
typedef unsigned v4u __attribute__((ext_vector_type(4)));
typedef int v8i __attribute__((ext_vector_type(8)));
typedef unsigned v16u __attribute__((ext_vector_type(16)));
typedef float v16f __attribute__((ext_vector_type(16)));
typedef unsigned long long u64;
#define LAS __attribute__((address_space(3)))
#define GAS __attribute__((address_space(1)))

constexpr int NRANK = 8;
constexpr int ML = 512;
constexpr int NN = 16384;
constexpr int KSL = 512;
constexpr int NSTEP = 32;
constexpr int NLOC = 4;
constexpr int DEPTH = 8;
constexpr int STAGE = 16384;
constexpr int HELD = 3;
constexpr int PUSHB = (DEPTH + 1) * STAGE;
constexpr int CTRL = PUSHB + 8192;
constexpr int LDS_BYTES = CTRL + 64;
constexpr unsigned BIG = 64;

constexpr unsigned XPIECE = 2048;
constexpr unsigned XBLK = 8 * XPIECE;
constexpr unsigned XSTEP = 8 * XBLK;
constexpr unsigned XSLOT = (NSTEP - NLOC) * XSTEP;
constexpr unsigned FLAG_OFF = 2 * XSLOT;
constexpr unsigned CNT_OFF = FLAG_OFF + 65536;
constexpr unsigned TMO_OFF = CNT_OFF + 32768;
constexpr unsigned DBG_OFF = TMO_OFF + 4096;
constexpr unsigned WS_NEED = DBG_OFF + (1u << 20);
constexpr u64 DEADLINE_TICKS = 50000000ull;
#define REP 32
#define OAUX 16
#define OAUX0 2
#define IDLE_FROM 0
#define IDLE_TO 32
#define PFIN 0
#define LPRIO 3
#define CPRIO 3
#define SWAP 1
#define TSW 32
#define SW2 8
#define POLL_SLEEP 1
#define PF_FIRST 5
#define LEAD 8
#define INFL0 1
#define INFL 2
#define PACE0 70
#define PACE1 300
#define LDELAY0 2
#define LDELAY1 14
#define XDELAY 8
#define PACE_ROUND 150
#define PACE_BLK 15

struct Args {
    const unsigned char* x;
    const unsigned char* w;
    const float* sx;
    const float* sw;
    float* out;
    int rank;
    int pad;
    unsigned char* ws[NRANK];
};

#define SB() __builtin_amdgcn_sched_barrier(0)
#define CFENCE() asm volatile("" ::: "memory")
#define NOW() __builtin_amdgcn_s_memrealtime()

#define TS(v) do {} while (0)

__device__ __forceinline__ v8i cat8(v4i lo, v4i hi) {
    return __builtin_shufflevector(lo, hi, 0, 1, 2, 3, 4, 5, 6, 7);
}

__device__ __forceinline__ v4i make_rsrc(const void* p, unsigned bytes) {
    const u64 q = (u64)p;
    v4i r;
    r.x = __builtin_amdgcn_readfirstlane((int)(unsigned)q);
    r.y = __builtin_amdgcn_readfirstlane((int)(unsigned)(q >> 32) & 0xffff);
    r.z = bytes;
    r.w = 0x00020000;
    return r;
}

__device__ __forceinline__ void dma16(unsigned m, unsigned voff, v4i rsrc, unsigned soff) {
    asm volatile("s_mov_b32 m0, %0\n\ts_nop 0\n\tbuffer_load_dwordx4 %1, %2, %3 offen lds"
                 :: "s"(m), "v"(voff), "s"(rsrc), "s"(soff) : "memory");
}

#define WPOL ""
__device__ __forceinline__ void dma16w(unsigned m, unsigned voff, v4i rsrc, unsigned soff) {
    asm volatile("s_mov_b32 m0, %0\n\ts_nop 0\n\tbuffer_load_dwordx4 %1, %2, %3 offen " WPOL " lds"
                 :: "s"(m), "v"(voff), "s"(rsrc), "s"(soff) : "memory");
}

__device__ __forceinline__ bool vm_idle() {
    unsigned r;
    asm volatile("s_getreg_b32 %0, hwreg(HW_REG_IB_STS)" : "=s"(r));
    return (r & 0xc000000fu) == 0;
}

__device__ __forceinline__ unsigned xlane_off(int lane) {
    const int g = lane >> 3;
    return (g + (g >= 3)) * 128 + (lane & 7) * 16;
}

__device__ __forceinline__ unsigned krow(int st, int rank) {
    if (st < NLOC) return rank * KSL + st * 128;
    const int t = st - NLOC, c = (t * 37) >> 8, i = t - c * 7;
    return ((rank + 1 + i) & 7) * KSL + c * 128;
}

__device__ __forceinline__ int kswap(int t, int h) { return t ^ (h * (t < TSW ? SWAP : SW2)); }

template <int V> using IC = std::integral_constant<int, V>;
template <int A, int B, class F> __device__ __forceinline__ void static_for(F&& f) {
    if constexpr (A < B) { f(IC<A>{}); static_for<A + 1, B>(f); }
}
constexpr unsigned xoff(int V) {
    return PFIN == 0 ? 0 : ((V % (2 * PFIN)) >> 1) * XSTEP + ((V & 1) * 4 + (V / (2 * PFIN)) * 2) * XPIECE;
}

__global__ void __launch_bounds__(512, 2) a2a_gemm(Args a) {
    extern __shared__ __attribute__((aligned(16))) unsigned char smem[];
    LAS unsigned char* L = (LAS unsigned char*)smem;
    const unsigned L0 = (unsigned)(size_t)L;
    const int tid = threadIdx.x;
    const int lane = tid & 63;
    const int wave = __builtin_amdgcn_readfirstlane(tid >> 6);
    const int b = blockIdx.x;
    const int rank = a.rank;
    const int idx = b >> 3;
    const int nt = (b & 7) * 16 + (idx >> 1);
    const int mh = idx & 1;
    const int n0 = nt * 128;
    unsigned char* wsl = a.ws[0];
    const unsigned lv0 = (lane >> 3) * KSL + (((lane & 7) ^ (lane >> 4)) << 4);
    const unsigned lv1 = lv0 ^ 64;
    const unsigned xrd = (lane & 31) * 128 + (((lane >> 5) ^ ((lane >> 1) & 7)) << 4);
    TS(t_start);
    if (tid < 12) ((volatile LAS unsigned*)(L + CTRL))[tid] = tid == 2 ? NLOC - 1 : 0u;
    asm volatile("s_waitcnt lgkmcnt(0)\n\ts_barrier" ::: "memory");
    TS(t_bar);

    if (wave == 7) {
        __builtin_amdgcn_s_setprio(CPRIO);
        const u64 t0 = NOW();
        const unsigned dj = (unsigned)(b >> 5) & (NRANK - 1);
        const int pmb = b & 7, c = (b >> 3) & 3;
        const bool self = dj == 0;
        GAS unsigned* cnt = (GAS unsigned*)(wsl + CNT_OFF) + b * 32;
        GAS unsigned* tmo = (GAS unsigned*)(wsl + TMO_OFF);
        const v4i rp = make_rsrc(a.x + (size_t)(((rank - dj) & (NRANK - 1)) * ML + pmb * 64) * KSL, 64 * KSL);
        auto fetch = [&]() {
#pragma unroll
            for (int q = 0; q < 8; ++q) dma16(L0 + PUSHB + q * 1024, (q & 1) ? lv1 : lv0, rp, q * 8 * KSL + c * 128);
        };
        asm volatile("s_nop 4" ::: "memory");
        int state = self ? 3 : 0;
        if (!self && c == 0) { fetch(); state = 1; }
        unsigned cv;
        asm volatile("s_load_dword %0, %1, 0 glc\n\ts_waitcnt lgkmcnt(0)" : "=&s"(cv) : "s"(cnt) : "memory");
        const unsigned call = cv + 1;
        volatile LAS unsigned* cw = (volatile LAS unsigned*)(L + CTRL);
        cw[3] = (call & 1) * XSLOT + 1;
        const int step = NLOC - 1 + 7 * c + (int)dj;
        const int t = kswap(step, pmb >> 2);
        const int ord = (pmb & 3) + (t > kswap(step, (pmb >> 2) ^ 1) ? 4 : 0);
        const u64 when = (c == 0 ? PACE0 : PACE1 + (c - 1) * PACE_ROUND) + ord * PACE_BLK;
        unsigned char* pw = a.ws[dj];
        unsigned char* dst = pw + (call & 1) * XSLOT + (self ? 0 : t - NLOC) * XSTEP + pmb * XBLK + xlane_off(lane);
        const unsigned char* fb = wsl + FLAG_OFF + (((b & 7) | ((idx >> 1) << 3)) & (REP - 1)) * 1024 + mh * 512;
        const unsigned char* fp = fb + (lane & 31) * 16;
        int ready = NLOC;
        for (unsigned spin = 1;; ++spin) {
            if (state < 3) {
                const u64 el = NOW() - t0;
                const bool idle = c == 0 || ready == NSTEP || el >= when + 400 || vm_idle();
                if (state == 0) {
                    if (el + 120 >= when) { fetch(); state = 1; }
                } else if (state == 1) {
                    if (el >= when && idle) {
                        asm volatile("s_waitcnt vmcnt(0)" ::: "memory");
                        v4u v[8];
#pragma unroll
                        for (int q = 0; q < 8; ++q)
                            v[q] = *(LAS v4u*)(L + PUSHB + (xrd ^ (((q >> 2) * 4 + (q & 1) * 2) << 4)) + ((q >> 1) & 1) * 4096);
#pragma unroll
                        for (int q = 0; q < 8; ++q)
                            asm volatile("global_store_dwordx4 %0, %1, off sc0 sc1\n\ts_nop 1" :: "v"(dst + q * XPIECE), "v"(v[q]) : "memory");
                        state = 2;
                        if (c == 0) continue;
                    }
                } else if (idle) {
                    asm volatile("s_waitcnt vmcnt(0)" ::: "memory");
                    if (lane < REP) {
                        GAS unsigned* f = (GAS unsigned*)(pw + FLAG_OFF + lane * 1024 + (pmb >> 2) * 512 + t * 16 + (pmb & 3) * 4);
                        __hip_atomic_store(f, call, __ATOMIC_RELAXED, __HIP_MEMORY_SCOPE_SYSTEM);
                    }
                    state = 3;
                }
                if (state < 3 && (ready == NSTEP || c == 0)) {
                    __builtin_amdgcn_s_sleep(1);
                    continue;
                }
            }
            if (ready == NSTEP) break;
            int r;
            if ((spin & 31) != 0) {
                const int w0 = ready & ~3;
                v16u g;
                asm volatile("s_load_dwordx16 %0, %1, 0 glc\n\ts_waitcnt lgkmcnt(0)" : "=&s"(g) : "s"(fb + w0 * 16) : "memory");
                unsigned m = 0;
#pragma unroll
                for (int q = 0; q < 4; ++q) {
                    unsigned lo = min(g[4 * q], g[4 * q + 1]), hi = min(g[4 * q + 2], g[4 * q + 3]);
                    asm volatile("" : "+s"(lo), "+s"(hi));
                    m |= (min(lo, hi) >= call ? 1u : 0u) << q;
                }
                r = w0 + __builtin_ctz(~m);
            } else {
                v4u f;
                asm volatile("global_load_dwordx4 %0, %1, off sc0 sc1\n\ts_waitcnt vmcnt(0)" : "=&v"(f) : "v"(fp) : "memory");
                const bool ok = min(min(f.x, f.y), min(f.z, f.w)) >= call;
                const unsigned mask = (unsigned)__builtin_amdgcn_ballot_w64(ok) | ((1u << NLOC) - 1);
                r = mask == 0xffffffffu ? NSTEP : __builtin_ctz(~mask);
            }
            if (r > ready) {
                cw[2] = r == NSTEP ? BIG : (unsigned)(r - 1);
                ready = r;
                continue;
            }
            if ((spin & 63) == 0
                && (__hip_atomic_load(tmo, __ATOMIC_RELAXED, __HIP_MEMORY_SCOPE_AGENT) == call
                    || NOW() - t0 > DEADLINE_TICKS)) {
                __hip_atomic_store(tmo, call, __ATOMIC_RELAXED, __HIP_MEMORY_SCOPE_AGENT);
                cw[8] = 1u;
                cw[2] = BIG;
                ready = NSTEP;
                continue;
            }
            __builtin_amdgcn_s_sleep(POLL_SLEEP);
        }
        if (lane == 0) __hip_atomic_store(cnt, call, __ATOMIC_RELAXED, __HIP_MEMORY_SCOPE_AGENT);
    } else if (wave < 4) {
        const int mb = mh * 4 + wave;
        const unsigned xreg = (DEPTH - wave) * STAGE;
        const v4i rl = make_rsrc(a.x + (size_t)(rank * ML + mb * 64) * KSL, 64 * KSL);
        auto xstage = [&](int st, int q) {
            dma16(L0 + xreg + (st & 1) * 8192 + q * 1024, (q & 1) ? lv1 : lv0, rl, q * 8 * KSL + (st ^ (mh * SWAP)) * 128);
        };
        __builtin_amdgcn_s_sleep(XDELAY);
#pragma unroll
        for (int q = 0; q < 16; ++q) xstage(q >> 3, q & 7);
        float scale;
        {
            float s0, s1;
            asm volatile("s_load_dword %0, %2, 0\n\ts_load_dword %1, %3, 0\n\ts_waitcnt lgkmcnt(0)"
                         : "=&s"(s0), "=&s"(s1) : "s"(a.sx), "s"(a.sw) : "memory");
            scale = __builtin_bit_cast(float, __builtin_amdgcn_readfirstlane(__builtin_bit_cast(int, s0 * s1)));
        }
        __builtin_amdgcn_s_setprio(2);
        v8i xf[4][2];
        v8i wf[4];
        v16f acc[2][4];
#pragma unroll
        for (int i = 0; i < 2; ++i)
#pragma unroll
            for (int j = 0; j < 4; ++j)
#pragma unroll
                for (int r = 0; r < 16; ++r) acc[i][j][r] = 0.f;
        const __amdgpu_buffer_rsrc_t rx = __builtin_amdgcn_make_buffer_rsrc((void*)wsl, 0, 2 * XSLOT, 0x00020000);
        unsigned voff = xlane_off(lane);
        float* obase = a.out + (size_t)mb * 64 * NN + n0;
        asm volatile("" : "+s"(scale), "+s"(obase));
        const __amdgpu_buffer_rsrc_t ro = __builtin_amdgcn_make_buffer_rsrc((void*)obase, 0, (64 * NN - n0) * 4, 0x00020000);
        unsigned wadr[4];
        {
            const int qq = (lane & 15) >> 1, p = lane & 1, cg = (lane >> 4) & 1, h = lane >> 5;
#pragma unroll
            for (int j = 0; j < 4; ++j)
                wadr[j] = (16 * h + qq) * 128 + ((j ^ ((qq >> 1) & 3)) << 5) + 16 * cg + 8 * p;
        }
        unsigned xadr = xreg + xrd;
        volatile LAS v4u* ctrl = (volatile LAS v4u*)(L + CTRL);
        volatile LAS unsigned* done = (volatile LAS unsigned*)(L + CTRL + 16) + wave;
        asm volatile("" : "+v"(acc[0][0]), "+v"(acc[0][1]), "+v"(acc[0][2]), "+v"(acc[0][3]),
                          "+v"(acc[1][0]), "+v"(acc[1][1]), "+v"(acc[1][2]), "+v"(acc[1][3]),
                          "+v"(wadr[0]), "+v"(wadr[1]), "+v"(wadr[2]), "+v"(wadr[3]), "+v"(voff), "+v"(xadr));
        auto xlds = [&](int u, int i) -> v8i {
            LAS unsigned char* p = L + (u >> 1) * 8192 + i * 4096;
            return cat8(*(LAS v4i*)(p + (xadr ^ ((u & 1) * 64))), *(LAS v4i*)(p + (xadr ^ ((u & 1) * 64 + 32))));
        };
        auto wread_at = [&](unsigned base, unsigned off) -> v8i {
            LAS unsigned char* p = L + base + off;
            v2i r0 = __builtin_amdgcn_ds_read_tr8_b64_v2i32((LAS v2i*)(p));
            v2i r1 = __builtin_amdgcn_ds_read_tr8_b64_v2i32((LAS v2i*)(p + 1024));
            v2i r2 = __builtin_amdgcn_ds_read_tr8_b64_v2i32((LAS v2i*)(p + 4096));
            v2i r3 = __builtin_amdgcn_ds_read_tr8_b64_v2i32((LAS v2i*)(p + 5120));
            return cat8(__builtin_shufflevector(r0, r1, 0, 1, 2, 3), __builtin_shufflevector(r2, r3, 0, 1, 2, 3));
        };
        auto wread = [&](int j, unsigned stage_off, int kk) -> v8i { return wread_at(stage_off + wadr[j], kk * 8192); };
        asm volatile("s_waitcnt vmcnt(8)" ::: "memory");
        TS(t_x);
        xf[0][0] = xlds(0, 0); xf[0][1] = xlds(0, 1);
        xf[1][0] = xlds(1, 0); xf[1][1] = xlds(1, 1);
        v4u cw = *ctrl;
        while (__builtin_amdgcn_readfirstlane(cw.x) < 1u) { __builtin_amdgcn_s_sleep(1); cw = *ctrl; }
        CFENCE();
        TS(t_w);
        wf[0] = wread(0, 0, 0);
        wf[1] = wread(1, 0, 0);
        wf[2] = wread(2, 0, 0);
        wf[3] = wf[2];
        xf[2][0] = xf[0][0]; xf[2][1] = xf[0][0]; xf[3][0] = xf[0][0]; xf[3][1] = xf[0][0];
        unsigned xs = 0;
        SB();
        auto sub = [&](auto uc, auto xc, auto lc, unsigned T, unsigned sc_, unsigned sn_) {
            constexpr int u = decltype(uc)::value, XS = decltype(xc)::value;
            constexpr bool LAST = decltype(lc)::value;
            v4i xp[4];
            if constexpr ((u & 1) == 1 && !LAST) {
                unsigned m = __builtin_amdgcn_readfirstlane(min(min(cw.x, cw.y), cw.z));
                if (__builtin_expect(m <= T + 1, 0)) {
                    while (m <= T + 1) {
                        __builtin_amdgcn_s_sleep(1);
                        cw = *ctrl;
                        m = __builtin_amdgcn_readfirstlane(min(min(cw.x, cw.y), cw.z));
                    }
                }
                CFENCE();
            }
            if constexpr (XS == 4 || XS == 5) asm volatile("s_waitcnt vmcnt(8)" ::: "memory");
#pragma unroll
            for (int j = 0; j < 4; ++j) {
                if (XS == 6 && j == 2) asm volatile("s_waitcnt vmcnt(0)" ::: "memory");
                if (XS == 2 && j == 1) {
                    if (__builtin_expect(__builtin_amdgcn_readfirstlane(cw.y) < 1u, 0))
                        do { cw = *ctrl; } while (__builtin_amdgcn_readfirstlane(cw.y) < 1u);
                    CFENCE();
                }
                acc[0][j] = __builtin_amdgcn_mfma_scale_f32_32x32x64_f8f6f4(xf[u][0], wf[j], acc[0][j], 0, 0, 0, 0, 0, 0);
                if constexpr (XS == 2 || XS == 3) {
                    SB();
                    if (XS == 3 && j == 0) asm volatile("s_waitcnt lgkmcnt(0)" ::: "memory");
                    xstage(XS, 2 * j);
                    SB();
                }
                acc[1][j] = __builtin_amdgcn_mfma_scale_f32_32x32x64_f8f6f4(xf[u][1], wf[j], acc[1][j], 0, 0, 0, 0, 0, 0);
                if (j == 0) wf[3] = wread(3, sc_, u & 1);
                else if (!LAST) wf[j - 1] = wread(j - 1, sn_, (u + 1) & 1);
                if constexpr (XS == 1) {
                    xp[j] = __builtin_amdgcn_raw_buffer_load_b128(rx, voff, xs + j * XPIECE, 17);
                    if (j == 1) xf[(u + 3) & 3][0] = cat8(xp[0], xp[1]);
                    if (j == 3) xf[(u + 3) & 3][1] = cat8(xp[2], xp[3]);
                    if (j == 3) xs += (u & 1) ? 4 * XPIECE : XSTEP - 4 * XPIECE;
                }
                if constexpr (XS == 7) {
                    constexpr int ub = (u + 3) & 3;
                    xp[j] = __builtin_amdgcn_raw_buffer_load_b128(rx, voff, xs + xoff(2 * ub + (j >> 1)) + (j & 1) * XPIECE, 17);
                    if (j == 1) xf[ub][0] = cat8(xp[0], xp[1]);
                    if (j == 3) xf[ub][1] = cat8(xp[2], xp[3]);
                }
                if constexpr (XS == 4) xf[2 + (j >> 1)][j & 1] = xlds(2 + (j >> 1), j & 1);
                if constexpr (XS == 5) xf[j >> 1][j & 1] = xlds(j >> 1, j & 1);
                if constexpr (XS == 6) { if (j >= 2) { xf[j][0] = xlds(j, 0); xf[j][1] = xlds(j, 1); } }
                if constexpr (XS == 2 || XS == 3) {
                    __builtin_amdgcn_sched_group_barrier(0x008, 1, 0);
                    __builtin_amdgcn_sched_group_barrier(0x100, 4, 0);
                    SB();
                    xstage(XS, 2 * j + 1);
                } else {
                    __builtin_amdgcn_sched_group_barrier(0x008, 1, 0);
                    __builtin_amdgcn_sched_group_barrier(0x100, XS >= 4 && XS <= 6 ? 4 : 2, 0);
                    __builtin_amdgcn_sched_group_barrier(0x020, 1, 0);
                    __builtin_amdgcn_sched_group_barrier(0x008, 1, 0);
                    __builtin_amdgcn_sched_group_barrier(0x100, XS >= 4 && XS <= 6 ? 4 : 2, 0);
                }
                SB();
                if ((u & 1) == 0 && j == 1) { cw = *ctrl; SB(); }
                if (((u & 1) == 1 && j == 0 && (XS <= 1 || XS == 7)) || (XS == 6 && j == 3)) {
                    CFENCE();
                    *done = (XS == 6 ? T : T + 1) + HELD;
                    CFENCE();
                    SB();
                }
            }
        };
        using NO = std::false_type; using YES = std::true_type;
#define HT(i)
        HT(0);
        sub(IC<0>{}, IC<2>{}, NO{}, 0, 0, 0);
        HT(1);
        sub(IC<1>{}, IC<4>{}, NO{}, 0, 0, STAGE);
        HT(2);
        sub(IC<2>{}, IC<3>{}, NO{}, 1, STAGE, STAGE);
        HT(3);
        sub(IC<3>{}, IC<5>{}, NO{}, 1, STAGE, 2 * STAGE);
        HT(4);
        sub(IC<0>{}, IC<6>{}, NO{}, 2, 2 * STAGE, 2 * STAGE);
        HT(5);
        if (__builtin_expect(__builtin_amdgcn_readfirstlane(cw.w) == 0u, 0))
            do { __builtin_amdgcn_s_sleep(1); cw = *ctrl; } while (__builtin_amdgcn_readfirstlane(cw.w) == 0u);
        xs = __builtin_amdgcn_readfirstlane(cw.w) - 1 + mb * XBLK;
        constexpr int TF = NSTEP - PFIN;
        for (unsigned T = 2; T < TF - 2; T += 2) {
            const unsigned so0 = (T & (DEPTH - 1)) * STAGE;
            const unsigned so2 = ((T + 2) & (DEPTH - 1)) * STAGE;
            sub(IC<1>{}, IC<1>{}, NO{}, T, so0, so0 + STAGE);
            sub(IC<2>{}, IC<1>{}, NO{}, T + 1, so0 + STAGE, so0 + STAGE);
            sub(IC<3>{}, IC<1>{}, NO{}, T + 1, so0 + STAGE, so2);
            sub(IC<0>{}, IC<1>{}, NO{}, T + 2, so2, so2);
        }
        constexpr auto slot = [](int T) { return (T & (DEPTH - 1)) * STAGE; };
        const unsigned vo = (4 * (lane >> 5) * NN + (lane & 31)) * 4;
        auto ostore = [&](int i, int k, float sc, auto auxc) {
            const int r = k >> 2, j = k & 3;
            __builtin_amdgcn_raw_buffer_store_b32(__builtin_bit_cast(unsigned, acc[i][j][r] * sc), ro, vo + j * 128,
                                                  (32 * i + (r & 3) + 8 * (r >> 2)) * NN * 4, decltype(auxc)::value);
        };
        float sc;
        if constexpr (PFIN == 0) {
            sub(IC<1>{}, IC<0>{}, NO{}, NSTEP - 2, 6 * STAGE, 7 * STAGE);
            sub(IC<2>{}, IC<0>{}, NO{}, NSTEP - 1, 7 * STAGE, 7 * STAGE);
            sub(IC<3>{}, IC<0>{}, YES{}, NSTEP - 1, 7 * STAGE, 7 * STAGE);
            sc = *(volatile LAS unsigned*)(L + CTRL + 32) ? __builtin_nanf("") : scale;
        } else {
            if (__builtin_expect(__builtin_amdgcn_readfirstlane(cw.z) < BIG, 0))
                do { __builtin_amdgcn_s_sleep(1); cw = *ctrl; } while (__builtin_amdgcn_readfirstlane(cw.z) < BIG);
            CFENCE();
            sub(IC<1>{}, IC<7>{}, NO{}, TF - 2, slot(TF - 2), slot(TF - 1));
            sub(IC<2>{}, IC<7>{}, NO{}, TF - 1, slot(TF - 1), slot(TF - 1));
            sub(IC<3>{}, IC<7>{}, NO{}, TF - 1, slot(TF - 1), slot(TF));
            sc = __builtin_bit_cast(float, __builtin_amdgcn_readfirstlane(__builtin_bit_cast(int,
                     *(volatile LAS unsigned*)(L + CTRL + 32) ? __builtin_nanf("") : scale)));
            unsigned wadh[4];
#pragma unroll
            for (int j = 0; j < 4; ++j) { wadh[j] = wadr[j] + 65536; asm volatile("" : "+v"(wadh[j]), "+v"(wadr[j])); }
            auto wfin = [&](int j, int w) -> v8i {
                const int o = slot(TF + w / 2) + (w & 1) * 8192;
                return wread_at(o >= 65536 ? wadh[j] : wadr[j], o & 65535);
            };
            SB();
            static_for<0, 4 * PFIN>([&](auto vc) {
                constexpr int V = decltype(vc)::value;
                constexpr int P2 = PFIN ? 2 * PFIN : 1, i = V / P2, w = V % P2, wn = (V + 1) % P2;
                if constexpr (i == 0 && (V & 1) == 1 && V + 1 < 2 * PFIN) {
                    unsigned m = __builtin_amdgcn_readfirstlane(min(cw.x, cw.y));
                    if (__builtin_expect(m <= TF + (V + 1) / 2, 0)) {
                        while (m <= TF + (V + 1) / 2) {
                            __builtin_amdgcn_s_sleep(1);
                            cw = *ctrl;
                            m = __builtin_amdgcn_readfirstlane(min(cw.x, cw.y));
                        }
                    }
                    CFENCE();
                }
                v4i xp[2];
#pragma unroll
                for (int j = 0; j < 4; ++j) {
                    acc[i][j] = __builtin_amdgcn_mfma_scale_f32_32x32x64_f8f6f4(xf[(V >> 1) & 3][V & 1], wf[j], acc[i][j], 0, 0, 0, 0, 0, 0);
                    if (j == 0) wf[3] = wfin(3, w);
                    else if (V + 1 < 4 * PFIN) wf[j - 1] = wfin(j - 1, wn);
                    if constexpr (V + 6 < 4 * PFIN) {
                        if (j & 1) xp[j >> 1] = __builtin_amdgcn_raw_buffer_load_b128(rx, voff, xs + xoff(V + 6) + (j >> 1) * XPIECE, 17);
                        if (j == 3) xf[((V + 6) >> 1) & 3][V & 1] = cat8(xp[0], xp[1]);
                    }
                    if constexpr (i == 1) {
                        constexpr int SPM = 8 / (PFIN ? PFIN : 1);
                        if (4 * w + j >= 1) {
#pragma unroll
                            for (int q = 0; q < SPM; ++q) ostore(0, (4 * w + j - 1) * SPM + q, sc, IC<OAUX0>{});
                        }
                    }
                    __builtin_amdgcn_sched_group_barrier(0x008, 1, 0);
                    __builtin_amdgcn_sched_group_barrier(0x100, 4, 0);
                    __builtin_amdgcn_sched_group_barrier(0x020, 1, 0);
                    SB();
                    if (i == 0 && (V & 1) == 0 && j == 0) { cw = *ctrl; SB(); }
                }
            });
        }
        TS(t_end);
        if constexpr (PFIN == 0) {
#pragma unroll
            for (int k = 0; k < 64; ++k) ostore(0, k, sc, IC<OAUX>{});
        } else {
#pragma unroll
            for (int k = 64 - 8 / (PFIN ? PFIN : 1); k < 64; ++k) ostore(0, k, sc, IC<OAUX>{});
        }
#pragma unroll
        for (int k = 0; k < 64; ++k) ostore(1, k, sc, IC<OAUX>{});
    } else if (wave < 6) {
        const int ld = wave - 4;
        const int rr = lane >> 3, p = lane & 7;
        const unsigned voff = rr * NN + ((((p >> 1) ^ ((rr >> 1) & 3)) << 5) | ((p & 1) << 4));
        const v4i rs = make_rsrc(a.w + n0, 4096 * NN - n0);
        auto issue = [&](int s) {
            const unsigned so = (krow(kswap(s, mh), rank) + ld * 64) * NN;
            const unsigned m = L0 + (s & (DEPTH - 1)) * STAGE + ld * 8192;
            __builtin_amdgcn_s_setprio(LPRIO);
#pragma unroll
            for (int q = 0; q < 8; ++q) dma16w(m + q * 1024, voff, rs, so + q * 8 * NN);
            __builtin_amdgcn_s_setprio(0);
        };
        if (ld == 0) __builtin_amdgcn_s_sleep(LDELAY0); else __builtin_amdgcn_s_sleep(LDELAY1);
        issue(0);
        TS(t_iss);
        volatile LAS unsigned* full = (volatile LAS unsigned*)(L + CTRL) + ld;
        volatile LAS v4u* done = (volatile LAS v4u*)(L + CTRL + 16);
        int issued = 1, pub = 0;
        while (pub < NSTEP) {
            if (issued < NSTEP && issued - pub < (pub == 0 ? INFL0 : INFL)) {
                const v4u d = *done;
                const int dm = __builtin_amdgcn_readfirstlane(min(min(d.x, d.y), min(d.z, d.w)));
                const int fr = max(dm + LEAD - HELD, DEPTH - HELD);
                CFENCE();
                if (issued < fr) { issue(issued); ++issued; continue; }
            }
            const int n = issued - pub;
            if (n == 0) { __builtin_amdgcn_s_sleep(2); continue; }
            switch (n) {
                case 1: asm volatile("s_waitcnt vmcnt(0)" ::: "memory"); break;
                case 2: asm volatile("s_waitcnt vmcnt(8)" ::: "memory"); break;
                case 3: asm volatile("s_waitcnt vmcnt(16)" ::: "memory"); break;
                default: asm volatile("s_waitcnt vmcnt(24)" ::: "memory"); break;
            }
            ++pub;
            *full = pub < NSTEP ? (unsigned)pub : BIG;
        }
    } else {
        {
            const unsigned char* px = a.x + (size_t)(rank * ML + mh * 256 + (idx >> 1) * 16) * KSL + lane * 64;
            unsigned t0_, t1_;
            asm volatile("global_load_dword %0, %2, off\n\tglobal_load_dword %1, %3, off\n\ts_waitcnt vmcnt(0)"
                         : "=&v"(t0_), "=&v"(t1_) : "v"(px), "v"(px + 4096) : "memory");
        }
        TS(t_xpf);
#define PF_DELAY 300
        { const u64 t0 = NOW(); while (NOW() - t0 < PF_DELAY) __builtin_amdgcn_s_sleep(8); }
        const unsigned char* wx = a.w + (b & 7) * 2048 + (size_t)(idx * 4) * NN;
        const unsigned l0 = (unsigned)((u64)wx >> 7);
        const unsigned o0 = ((3 - l0) & 7) * 128, o1 = o0 + 1024;
        const unsigned e = (63 - l0) & 63;
        const unsigned o2 = e < 16 ? e * 128 : o1;
        unsigned dmy = 0;
        for (int s = PF_FIRST; s < NSTEP; ++s) {
            const int sq = s < TSW ? s : TSW + ((s - TSW) / (2 * SW2)) * 2 * SW2 + (((s - TSW) % (2 * SW2)) >> 1) + ((s - TSW) & 1) * SW2;
            const unsigned char* p = wx + (size_t)krow(sq, rank) * NN;
            const unsigned char* p0 = p + o0;
            const unsigned char* p1 = p + o1;
#pragma unroll
            for (int q = 0; q < 8; ++q) {
                asm volatile("s_load_dword %0, %1, %2" : "+s"(dmy) : "s"(p0), "i"((q >> 1) * NN + (q & 1) * 64) : "memory");
                asm volatile("s_load_dword %0, %1, %2" : "+s"(dmy) : "s"(p1), "i"((q >> 1) * NN + (q & 1) * 64) : "memory");
            }
            if (e < 16) {
                const unsigned char* p2 = p + o2;
#pragma unroll
                for (int q = 0; q < 8; ++q)
                    asm volatile("s_load_dword %0, %1, %2" : "+s"(dmy) : "s"(p2), "i"((q >> 1) * NN + (q & 1) * 64) : "memory");
            }
        }
        asm volatile("s_waitcnt lgkmcnt(0)" : "+s"(dmy) :: "memory");
    }
}

extern "C" void kernel_launch(int rank, int nranks, ncclComm_t comm,
                              hipStream_t stream,
                              void* const* d_in, const int* in_sizes, int n_in,
                              void* d_out, int out_size,
                              const int* in_dtypes, int out_dtype,
                              void* const* d_out_all,
                              void* const* d_ws_all, size_t ws_bytes) {
    if (nranks != NRANK || (unsigned)rank >= (unsigned)NRANK || n_in < 4 || ws_bytes < WS_NEED
        || out_size != ML * NN || in_sizes[0] != 4096 * KSL || in_sizes[1] != 4096 * NN) {
        fprintf(stderr, "unexpected problem shape\n");
        return;
    }
    static bool init = false;
    if (!init) {
        HIP_CHECK(hipFuncSetAttribute((const void*)a2a_gemm, hipFuncAttributeMaxDynamicSharedMemorySize, LDS_BYTES));
        init = true;
    }
    Args a;
    a.x = (const unsigned char*)d_in[0];
    a.w = (const unsigned char*)d_in[1];
    a.sx = (const float*)d_in[2];
    a.sw = (const float*)d_in[3];
    a.out = (float*)d_out;
    a.rank = rank;
    a.pad = 0;
    for (int j = 0; j < NRANK; ++j) a.ws[j] = (unsigned char*)d_ws_all[(rank - j) & (NRANK - 1)];
    hipLaunchKernelGGL(a2a_gemm, dim3(256), dim3(512), LDS_BYTES, stream, a);
    HIP_CHECK(hipGetLastError());
}
```

```cpp
#include <hip/hip_runtime.h>
#include <rccl/rccl.h>
#include <cstdio>
#include <type_traits>

#define HIP_CHECK(expr) do { hipError_t _e = (expr); if (_e != hipSuccess) { \
    fprintf(stderr, "HIP error %s at %s:%d\n", hipGetErrorString(_e), __FILE__, __LINE__); \
    return; } } while (0)

typedef int v2i __attribute__((ext_vector_type(2)));
typedef int v4i __attribute__((ext_vector_type(4)));
typedef unsigned v4u __attribute__((ext_vector_type(4)));
typedef int v8i __attribute__((ext_vector_type(8)));
typedef unsigned v16u __attribute__((ext_vector_type(16)));
typedef float v16f __attribute__((ext_vector_type(16)));
typedef unsigned long long u64;
#define LAS __attribute__((address_space(3)))
#define GAS __attribute__((address_space(1)))

constexpr int NRANK = 8;
constexpr int ML = 512;
constexpr int NN = 16384;
constexpr int KSL = 512;
constexpr int NSTEP = 32;
constexpr int NLOC = 4;
constexpr int DEPTH = 8;
constexpr int STAGE = 16384;
constexpr int HELD = 3;
constexpr int PUSHB = (DEPTH + 1) * STAGE;
constexpr int CTRL = PUSHB + 8192;
constexpr int LDS_BYTES = CTRL + 64;
constexpr unsigned BIG = 64;

constexpr unsigned XPIECE = 2048;
constexpr unsigned XBLK = 8 * XPIECE;
constexpr unsigned XSTEP = 8 * XBLK;
constexpr unsigned XSLOT = (NSTEP - NLOC) * XSTEP;
constexpr unsigned FLAG_OFF = 2 * XSLOT;
constexpr unsigned CNT_OFF = FLAG_OFF + 65536;
constexpr unsigned TMO_OFF = CNT_OFF + 32768;
constexpr unsigned DBG_OFF = TMO_OFF + 4096;
constexpr unsigned WS_NEED = DBG_OFF + (1u << 20);
constexpr u64 DEADLINE_TICKS = 50000000ull;
#define REP 32
#define OAUX 16
#define OAUX0 16
#define IDLE_FROM 0
#define IDLE_TO 32
#define PFIN 0
#define SPM (8 / (PFIN ? PFIN : 1))
#define LPRIO 3
#define CPRIO 3
#define SWAP 1
#define TSW 32
#define SW2 8
#define POLL_SLEEP 1
#define PF_FIRST 5
#define LEAD 8
#define INFL0 1
#define INFL 2
#define PACE0 70
#define PACE1 300
#define LDELAY0 2
#define LDELAY1 14
#define XDELAY 8
#define PACE_ROUND 150
#define PACE_BLK 15

struct Args {
    const unsigned char* x;
    const unsigned char* w;
    const float* sx;
    const float* sw;
    float* out;
    int rank;
    int pad;
    unsigned char* ws[NRANK];
};

#define SB() __builtin_amdgcn_sched_barrier(0)
#define CFENCE() asm volatile("" ::: "memory")
#define NOW() __builtin_amdgcn_s_memrealtime()

#define TS(v) do {} while (0)

__device__ __forceinline__ v8i cat8(v4i lo, v4i hi) {
    return __builtin_shufflevector(lo, hi, 0, 1, 2, 3, 4, 5, 6, 7);
}

__device__ __forceinline__ v4i make_rsrc(const void* p, unsigned bytes) {
    const u64 q = (u64)p;
    v4i r;
    r.x = __builtin_amdgcn_readfirstlane((int)(unsigned)q);
    r.y = __builtin_amdgcn_readfirstlane((int)(unsigned)(q >> 32) & 0xffff);
    r.z = bytes;
    r.w = 0x00020000;
    return r;
}

__device__ __forceinline__ void dma16(unsigned m, unsigned voff, v4i rsrc, unsigned soff) {
    asm volatile("s_mov_b32 m0, %0\n\ts_nop 0\n\tbuffer_load_dwordx4 %1, %2, %3 offen lds"
                 :: "s"(m), "v"(voff), "s"(rsrc), "s"(soff) : "memory");
}

#define WPOL ""
__device__ __forceinline__ void dma16w(unsigned m, unsigned voff, v4i rsrc, unsigned soff) {
    asm volatile("s_mov_b32 m0, %0\n\ts_nop 0\n\tbuffer_load_dwordx4 %1, %2, %3 offen " WPOL " lds"
                 :: "s"(m), "v"(voff), "s"(rsrc), "s"(soff) : "memory");
}

__device__ __forceinline__ bool vm_idle() {
    unsigned r;
    asm volatile("s_getreg_b32 %0, hwreg(HW_REG_IB_STS)" : "=s"(r));
    return (r & 0xc000000fu) == 0;
}

__device__ __forceinline__ unsigned xlane_off(int lane) {
    const int g = lane >> 3;
    return (g + (g >= 3)) * 128 + (lane & 7) * 16;
}

__device__ __forceinline__ unsigned krow(int st, int rank) {
    if (st < NLOC) return rank * KSL + st * 128;
    const int t = st - NLOC, c = (t * 37) >> 8, i = t - c * 7;
    return ((rank + 1 + i) & 7) * KSL + c * 128;
}

__device__ __forceinline__ int kswap(int t, int h) { return t ^ (h * (t < TSW ? SWAP : SW2)); }

template <int V> using IC = std::integral_constant<int, V>;
template <int A, int B, class F> __device__ __forceinline__ void static_for(F&& f) {
    if constexpr (A < B) { f(IC<A>{}); static_for<A + 1, B>(f); }
}
constexpr unsigned xoff(int V) {
    return PFIN == 0 ? 0 : ((V % (2 * PFIN)) >> 1) * XSTEP + ((V & 1) * 4 + (V / (2 * PFIN)) * 2) * XPIECE;
}

__global__ void __launch_bounds__(512, 2) a2a_gemm(Args a) {
    extern __shared__ __attribute__((aligned(16))) unsigned char smem[];
    LAS unsigned char* L = (LAS unsigned char*)smem;
    const unsigned L0 = (unsigned)(size_t)L;
    const int tid = threadIdx.x;
    const int lane = tid & 63;
    const int wave = __builtin_amdgcn_readfirstlane(tid >> 6);
    const int b = blockIdx.x;
    const int rank = a.rank;
    const int idx = b >> 3;
    const int nt = (b & 7) * 16 + (idx >> 1);
    const int mh = idx & 1;
    const int n0 = nt * 128;
    unsigned char* wsl = a.ws[0];
    const unsigned lv0 = (lane >> 3) * KSL + (((lane & 7) ^ (lane >> 4)) << 4);
    const unsigned lv1 = lv0 ^ 64;
    const unsigned xrd = (lane & 31) * 128 + (((lane >> 5) ^ ((lane >> 1) & 7)) << 4);
    TS(t_start);
    if (tid < 12) ((volatile LAS unsigned*)(L + CTRL))[tid] = tid == 2 ? NLOC - 1 : 0u;
    asm volatile("s_waitcnt lgkmcnt(0)\n\ts_barrier" ::: "memory");
    TS(t_bar);

    if (wave == 7) {
        __builtin_amdgcn_s_setprio(CPRIO);
        const u64 t0 = NOW();
        const unsigned dj = (unsigned)(b >> 5) & (NRANK - 1);
        const int pmb = b & 7, c = (b >> 3) & 3;
        const bool self = dj == 0;
        GAS unsigned* cnt = (GAS unsigned*)(wsl + CNT_OFF) + b * 32;
        GAS unsigned* tmo = (GAS unsigned*)(wsl + TMO_OFF);
        const v4i rp = make_rsrc(a.x + (size_t)(((rank - dj) & (NRANK - 1)) * ML + pmb * 64) * KSL, 64 * KSL);
        auto fetch = [&]() {
#pragma unroll
            for (int q = 0; q < 8; ++q) dma16(L0 + PUSHB + q * 1024, (q & 1) ? lv1 : lv0, rp, q * 8 * KSL + c * 128);
        };
        asm volatile("s_nop 4" ::: "memory");
        int state = self ? 3 : 0;
        if (!self && c == 0) { fetch(); state = 1; }
        unsigned cv;
        asm volatile("s_load_dword %0, %1, 0 glc\n\ts_waitcnt lgkmcnt(0)" : "=&s"(cv) : "s"(cnt) : "memory");
        const unsigned call = cv + 1;
        volatile LAS unsigned* cw = (volatile LAS unsigned*)(L + CTRL);
        cw[3] = (call & 1) * XSLOT + 1;
        const int step = NLOC - 1 + 7 * c + (int)dj;
        const int t = kswap(step, pmb >> 2);
        const int ord = (pmb & 3) + (t > kswap(step, (pmb >> 2) ^ 1) ? 4 : 0);
        const u64 when = (c == 0 ? PACE0 : PACE1 + (c - 1) * PACE_ROUND) + ord * PACE_BLK;
        unsigned char* pw = a.ws[dj];
        unsigned char* dst = pw + (call & 1) * XSLOT + (self ? 0 : t - NLOC) * XSTEP + pmb * XBLK + xlane_off(lane);
        const unsigned char* fb = wsl + FLAG_OFF + (((b & 7) | ((idx >> 1) << 3)) & (REP - 1)) * 1024 + mh * 512;
        const unsigned char* fp = fb + (lane & 31) * 16;
        int ready = NLOC;
        for (unsigned spin = 1;; ++spin) {
            if (state < 3) {
                const u64 el = NOW() - t0;
                const bool idle = c == 0 || ready == NSTEP || el >= when + 400 || vm_idle();
                if (state == 0) {
                    if (el + 120 >= when) { fetch(); state = 1; }
                } else if (state == 1) {
                    if (el >= when && idle) {
                        asm volatile("s_waitcnt vmcnt(0)" ::: "memory");
                        v4u v[8];
#pragma unroll
                        for (int q = 0; q < 8; ++q)
                            v[q] = *(LAS v4u*)(L + PUSHB + (xrd ^ (((q >> 2) * 4 + (q & 1) * 2) << 4)) + ((q >> 1) & 1) * 4096);
#pragma unroll
                        for (int q = 0; q < 8; ++q)
                            asm volatile("global_store_dwordx4 %0, %1, off sc0 sc1\n\ts_nop 1" :: "v"(dst + q * XPIECE), "v"(v[q]) : "memory");
                        state = 2;
                        if (c == 0) continue;
                    }
                } else if (idle) {
                    asm volatile("s_waitcnt vmcnt(0)" ::: "memory");
                    if (lane < REP) {
                        GAS unsigned* f = (GAS unsigned*)(pw + FLAG_OFF + lane * 1024 + (pmb >> 2) * 512 + t * 16 + (pmb & 3) * 4);
                        __hip_atomic_store(f, call, __ATOMIC_RELAXED, __HIP_MEMORY_SCOPE_SYSTEM);
                    }
                    state = 3;
                }
                if (state < 3 && (ready == NSTEP || c == 0)) {
                    __builtin_amdgcn_s_sleep(1);
                    continue;
                }
            }
            if (ready == NSTEP) break;
            int r;
            if ((spin & 31) != 0) {
                const int w0 = ready & ~3;
                v16u g;
                asm volatile("s_load_dwordx16 %0, %1, 0 glc\n\ts_waitcnt lgkmcnt(0)" : "=&s"(g) : "s"(fb + w0 * 16) : "memory");
                unsigned m = 0;
#pragma unroll
                for (int q = 0; q < 4; ++q) {
                    unsigned lo = min(g[4 * q], g[4 * q + 1]), hi = min(g[4 * q + 2], g[4 * q + 3]);
                    asm volatile("" : "+s"(lo), "+s"(hi));
                    m |= (min(lo, hi) >= call ? 1u : 0u) << q;
                }
                r = w0 + __builtin_ctz(~m);
            } else {
                v4u f;
                asm volatile("global_load_dwordx4 %0, %1, off sc0 sc1\n\ts_waitcnt vmcnt(0)" : "=&v"(f) : "v"(fp) : "memory");
                const bool ok = min(min(f.x, f.y), min(f.z, f.w)) >= call;
                const unsigned mask = (unsigned)__builtin_amdgcn_ballot_w64(ok) | ((1u << NLOC) - 1);
                r = mask == 0xffffffffu ? NSTEP : __builtin_ctz(~mask);
            }
            if (r > ready) {
                cw[2] = r == NSTEP ? BIG : (unsigned)(r - 1);
                ready = r;
                continue;
            }
            if ((spin & 63) == 0
                && (__hip_atomic_load(tmo, __ATOMIC_RELAXED, __HIP_MEMORY_SCOPE_AGENT) == call
                    || NOW() - t0 > DEADLINE_TICKS)) {
                __hip_atomic_store(tmo, call, __ATOMIC_RELAXED, __HIP_MEMORY_SCOPE_AGENT);
                cw[8] = 1u;
                cw[2] = BIG;
                ready = NSTEP;
                continue;
            }
            __builtin_amdgcn_s_sleep(POLL_SLEEP);
        }
        if (lane == 0) __hip_atomic_store(cnt, call, __ATOMIC_RELAXED, __HIP_MEMORY_SCOPE_AGENT);
    } else if (wave < 4) {
        const int mb = mh * 4 + wave;
        const unsigned xreg = (DEPTH - wave) * STAGE;
        const v4i rl = make_rsrc(a.x + (size_t)(rank * ML + mb * 64) * KSL, 64 * KSL);
        auto xstage = [&](int st, int q) {
            dma16(L0 + xreg + (st & 1) * 8192 + q * 1024, (q & 1) ? lv1 : lv0, rl, q * 8 * KSL + (st ^ (mh * SWAP)) * 128);
        };
        __builtin_amdgcn_s_sleep(XDELAY);
#pragma unroll
        for (int q = 0; q < 16; ++q) xstage(q >> 3, q & 7);
        float scale;
        {
            float s0, s1;
            asm volatile("s_load_dword %0, %2, 0\n\ts_load_dword %1, %3, 0\n\ts_waitcnt lgkmcnt(0)"
                         : "=&s"(s0), "=&s"(s1) : "s"(a.sx), "s"(a.sw) : "memory");
            scale = __builtin_bit_cast(float, __builtin_amdgcn_readfirstlane(__builtin_bit_cast(int, s0 * s1)));
        }
        __builtin_amdgcn_s_setprio(2);
        v8i xf[4][2];
        v8i wf[4];
        v16f acc[2][4];
#pragma unroll
        for (int i = 0; i < 2; ++i)
#pragma unroll
            for (int j = 0; j < 4; ++j)
#pragma unroll
                for (int r = 0; r < 16; ++r) acc[i][j][r] = 0.f;
        const __amdgpu_buffer_rsrc_t rx = __builtin_amdgcn_make_buffer_rsrc((void*)wsl, 0, 2 * XSLOT, 0x00020000);
        unsigned voff = xlane_off(lane);
        float* obase = a.out + (size_t)mb * 64 * NN + n0;
        asm volatile("" : "+s"(scale), "+s"(obase));
        const __amdgpu_buffer_rsrc_t ro = __builtin_amdgcn_make_buffer_rsrc((void*)obase, 0, (64 * NN - n0) * 4, 0x00020000);
        unsigned wadr[4];
        {
            const int qq = (lane & 15) >> 1, p = lane & 1, cg = (lane >> 4) & 1, h = lane >> 5;
#pragma unroll
            for (int j = 0; j < 4; ++j)
                wadr[j] = (16 * h + qq) * 128 + ((j ^ ((qq >> 1) & 3)) << 5) + 16 * cg + 8 * p;
        }
        unsigned xadr = xreg + xrd;
        volatile LAS v4u* ctrl = (volatile LAS v4u*)(L + CTRL);
        volatile LAS unsigned* done = (volatile LAS unsigned*)(L + CTRL + 16) + wave;
        asm volatile("" : "+v"(acc[0][0]), "+v"(acc[0][1]), "+v"(acc[0][2]), "+v"(acc[0][3]),
                          "+v"(acc[1][0]), "+v"(acc[1][1]), "+v"(acc[1][2]), "+v"(acc[1][3]),
                          "+v"(wadr[0]), "+v"(wadr[1]), "+v"(wadr[2]), "+v"(wadr[3]), "+v"(voff), "+v"(xadr));
        auto xlds = [&](int u, int i) -> v8i {
            LAS unsigned char* p = L + (u >> 1) * 8192 + i * 4096;
            return cat8(*(LAS v4i*)(p + (xadr ^ ((u & 1) * 64))), *(LAS v4i*)(p + (xadr ^ ((u & 1) * 64 + 32))));
        };
        auto wread_at = [&](unsigned base, unsigned off) -> v8i {
            LAS unsigned char* p = L + base + off;
            v2i r0 = __builtin_amdgcn_ds_read_tr8_b64_v2i32((LAS v2i*)(p));
            v2i r1 = __builtin_amdgcn_ds_read_tr8_b64_v2i32((LAS v2i*)(p + 1024));
            v2i r2 = __builtin_amdgcn_ds_read_tr8_b64_v2i32((LAS v2i*)(p + 4096));
            v2i r3 = __builtin_amdgcn_ds_read_tr8_b64_v2i32((LAS v2i*)(p + 5120));
            return cat8(__builtin_shufflevector(r0, r1, 0, 1, 2, 3), __builtin_shufflevector(r2, r3, 0, 1, 2, 3));
        };
        auto wread = [&](int j, unsigned stage_off, int kk) -> v8i { return wread_at(stage_off + wadr[j], kk * 8192); };
        asm volatile("s_waitcnt vmcnt(8)" ::: "memory");
        TS(t_x);
        xf[0][0] = xlds(0, 0); xf[0][1] = xlds(0, 1);
        xf[1][0] = xlds(1, 0); xf[1][1] = xlds(1, 1);
        v4u cw = *ctrl;
        while (__builtin_amdgcn_readfirstlane(cw.x) < 1u) { __builtin_amdgcn_s_sleep(1); cw = *ctrl; }
        CFENCE();
        TS(t_w);
        wf[0] = wread(0, 0, 0);
        wf[1] = wread(1, 0, 0);
        wf[2] = wread(2, 0, 0);
        wf[3] = wf[2];
        xf[2][0] = xf[0][0]; xf[2][1] = xf[0][0]; xf[3][0] = xf[0][0]; xf[3][1] = xf[0][0];
        unsigned xs = 0;
        SB();
        auto sub = [&](auto uc, auto xc, auto lc, unsigned T, unsigned sc_, unsigned sn_) {
            constexpr int u = decltype(uc)::value, XS = decltype(xc)::value;
            constexpr bool LAST = decltype(lc)::value;
            v4i xp[4];
            if constexpr ((u & 1) == 1 && !LAST) {
                unsigned m = __builtin_amdgcn_readfirstlane(min(min(cw.x, cw.y), cw.z));
                if (__builtin_expect(m <= T + 1, 0)) {
                    while (m <= T + 1) {
                        __builtin_amdgcn_s_sleep(1);
                        cw = *ctrl;
                        m = __builtin_amdgcn_readfirstlane(min(min(cw.x, cw.y), cw.z));
                    }
                }
                CFENCE();
            }
            if constexpr (XS == 4 || XS == 5) asm volatile("s_waitcnt vmcnt(8)" ::: "memory");
#pragma unroll
            for (int j = 0; j < 4; ++j) {
                if (XS == 6 && j == 2) asm volatile("s_waitcnt vmcnt(0)" ::: "memory");
                if (XS == 2 && j == 1) {
                    if (__builtin_expect(__builtin_amdgcn_readfirstlane(cw.y) < 1u, 0))
                        do { cw = *ctrl; } while (__builtin_amdgcn_readfirstlane(cw.y) < 1u);
                    CFENCE();
                }
                acc[0][j] = __builtin_amdgcn_mfma_scale_f32_32x32x64_f8f6f4(xf[u][0], wf[j], acc[0][j], 0, 0, 0, 0, 0, 0);
                if constexpr (XS == 2 || XS == 3) {
                    SB();
                    if (XS == 3 && j == 0) asm volatile("s_waitcnt lgkmcnt(0)" ::: "memory");
                    xstage(XS, 2 * j);
                    SB();
                }
                acc[1][j] = __builtin_amdgcn_mfma_scale_f32_32x32x64_f8f6f4(xf[u][1], wf[j], acc[1][j], 0, 0, 0, 0, 0, 0);
                if (j == 0) wf[3] = wread(3, sc_, u & 1);
                else if (!LAST) wf[j - 1] = wread(j - 1, sn_, (u + 1) & 1);
                if constexpr (XS == 1) {
                    xp[j] = __builtin_amdgcn_raw_buffer_load_b128(rx, voff, xs + j * XPIECE, 17);
                    if (j == 1) xf[(u + 3) & 3][0] = cat8(xp[0], xp[1]);
                    if (j == 3) xf[(u + 3) & 3][1] = cat8(xp[2], xp[3]);
                    if (j == 3) xs += (u & 1) ? 4 * XPIECE : XSTEP - 4 * XPIECE;
                }
                if constexpr (XS == 7) {
                    constexpr int ub = (u + 3) & 3;
                    xp[j] = __builtin_amdgcn_raw_buffer_load_b128(rx, voff, xs + xoff(2 * ub + (j >> 1)) + (j & 1) * XPIECE, 17);
                    if (j == 1) xf[ub][0] = cat8(xp[0], xp[1]);
                    if (j == 3) xf[ub][1] = cat8(xp[2], xp[3]);
                }
                if constexpr (XS == 4) xf[2 + (j >> 1)][j & 1] = xlds(2 + (j >> 1), j & 1);
                if constexpr (XS == 5) xf[j >> 1][j & 1] = xlds(j >> 1, j & 1);
                if constexpr (XS == 6) { if (j >= 2) { xf[j][0] = xlds(j, 0); xf[j][1] = xlds(j, 1); } }
                if constexpr (XS == 2 || XS == 3) {
                    __builtin_amdgcn_sched_group_barrier(0x008, 1, 0);
                    __builtin_amdgcn_sched_group_barrier(0x100, 4, 0);
                    SB();
                    xstage(XS, 2 * j + 1);
                } else {
                    __builtin_amdgcn_sched_group_barrier(0x008, 1, 0);
                    __builtin_amdgcn_sched_group_barrier(0x100, XS >= 4 && XS <= 6 ? 4 : 2, 0);
                    __builtin_amdgcn_sched_group_barrier(0x020, 1, 0);
                    __builtin_amdgcn_sched_group_barrier(0x008, 1, 0);
                    __builtin_amdgcn_sched_group_barrier(0x100, XS >= 4 && XS <= 6 ? 4 : 2, 0);
                }
                SB();
                if ((u & 1) == 0 && j == 1) { cw = *ctrl; SB(); }
                if (((u & 1) == 1 && j == 0 && (XS <= 1 || XS == 7)) || (XS == 6 && j == 3)) {
                    CFENCE();
                    *done = (XS == 6 ? T : T + 1) + HELD;
                    CFENCE();
                    SB();
                }
            }
        };
        using NO = std::false_type; using YES = std::true_type;
#define HT(i)
        HT(0);
        sub(IC<0>{}, IC<2>{}, NO{}, 0, 0, 0);
        HT(1);
        sub(IC<1>{}, IC<4>{}, NO{}, 0, 0, STAGE);
        HT(2);
        sub(IC<2>{}, IC<3>{}, NO{}, 1, STAGE, STAGE);
        HT(3);
        sub(IC<3>{}, IC<5>{}, NO{}, 1, STAGE, 2 * STAGE);
        HT(4);
        sub(IC<0>{}, IC<6>{}, NO{}, 2, 2 * STAGE, 2 * STAGE);
        HT(5);
        if (__builtin_expect(__builtin_amdgcn_readfirstlane(cw.w) == 0u, 0))
            do { __builtin_amdgcn_s_sleep(1); cw = *ctrl; } while (__builtin_amdgcn_readfirstlane(cw.w) == 0u);
        xs = __builtin_amdgcn_readfirstlane(cw.w) - 1 + mb * XBLK;
        constexpr int TF = NSTEP - PFIN;
        for (unsigned T = 2; T < TF - 2; T += 2) {
            const unsigned so0 = (T & (DEPTH - 1)) * STAGE;
            const unsigned so2 = ((T + 2) & (DEPTH - 1)) * STAGE;
            sub(IC<1>{}, IC<1>{}, NO{}, T, so0, so0 + STAGE);
            sub(IC<2>{}, IC<1>{}, NO{}, T + 1, so0 + STAGE, so0 + STAGE);
            sub(IC<3>{}, IC<1>{}, NO{}, T + 1, so0 + STAGE, so2);
            sub(IC<0>{}, IC<1>{}, NO{}, T + 2, so2, so2);
        }
        constexpr auto slot = [](int T) { return (T & (DEPTH - 1)) * STAGE; };
        const unsigned vo = (4 * (lane >> 5) * NN + (lane & 31)) * 4;
        auto ostore = [&](int i, int k, float sc, auto auxc) {
            const int r = k >> 2, j = k & 3;
            __builtin_amdgcn_raw_buffer_store_b32(__builtin_bit_cast(unsigned, acc[i][j][r] * sc), ro, vo + j * 128,
                                                  (32 * i + (r & 3) + 8 * (r >> 2)) * NN * 4, decltype(auxc)::value);
        };
        float sc;
        if constexpr (PFIN == 0) {
            sub(IC<1>{}, IC<0>{}, NO{}, NSTEP - 2, 6 * STAGE, 7 * STAGE);
            sub(IC<2>{}, IC<0>{}, NO{}, NSTEP - 1, 7 * STAGE, 7 * STAGE);
            sub(IC<3>{}, IC<0>{}, YES{}, NSTEP - 1, 7 * STAGE, 7 * STAGE);
            sc = *(volatile LAS unsigned*)(L + CTRL + 32) ? __builtin_nanf("") : scale;
        } else {
            if (__builtin_expect(__builtin_amdgcn_readfirstlane(cw.z) < BIG, 0))
                do { __builtin_amdgcn_s_sleep(1); cw = *ctrl; } while (__builtin_amdgcn_readfirstlane(cw.z) < BIG);
            CFENCE();
            sub(IC<1>{}, IC<7>{}, NO{}, TF - 2, slot(TF - 2), slot(TF - 1));
            sub(IC<2>{}, IC<7>{}, NO{}, TF - 1, slot(TF - 1), slot(TF - 1));
            sub(IC<3>{}, IC<7>{}, NO{}, TF - 1, slot(TF - 1), slot(TF));
            sc = __builtin_bit_cast(float, __builtin_amdgcn_readfirstlane(__builtin_bit_cast(int,
                     *(volatile LAS unsigned*)(L + CTRL + 32) ? __builtin_nanf("") : scale)));
            unsigned wadh[4];
#pragma unroll
            for (int j = 0; j < 4; ++j) { wadh[j] = wadr[j] + 65536; asm volatile("" : "+v"(wadh[j]), "+v"(wadr[j])); }
            auto wfin = [&](int j, int w) -> v8i {
                const int o = slot(TF + w / 2) + (w & 1) * 8192;
                return wread_at(o >= 65536 ? wadh[j] : wadr[j], o & 65535);
            };
            SB();
            static_for<0, 4 * PFIN>([&](auto vc) {
                constexpr int V = decltype(vc)::value;
                constexpr int P2 = PFIN ? 2 * PFIN : 1, i = V / P2, w = V % P2, wn = (V + 1) % P2;
                if constexpr (i == 0 && (V & 1) == 1 && V + 1 < 2 * PFIN) {
                    unsigned m = __builtin_amdgcn_readfirstlane(min(cw.x, cw.y));
                    if (__builtin_expect(m <= TF + (V + 1) / 2, 0)) {
                        while (m <= TF + (V + 1) / 2) {
                            __builtin_amdgcn_s_sleep(1);
                            cw = *ctrl;
                            m = __builtin_amdgcn_readfirstlane(min(cw.x, cw.y));
                        }
                    }
                    CFENCE();
                }
                v4i xp[2];
#pragma unroll
                for (int j = 0; j < 4; ++j) {
                    acc[i][j] = __builtin_amdgcn_mfma_scale_f32_32x32x64_f8f6f4(xf[(V >> 1) & 3][V & 1], wf[j], acc[i][j], 0, 0, 0, 0, 0, 0);
                    if (j == 0) wf[3] = wfin(3, w);
                    else if (V + 1 < 4 * PFIN) wf[j - 1] = wfin(j - 1, wn);
                    if constexpr (V + 6 < 4 * PFIN) {
                        if (j & 1) xp[j >> 1] = __builtin_amdgcn_raw_buffer_load_b128(rx, voff, xs + xoff(V + 6) + (j >> 1) * XPIECE, 17);
                        if (j == 3) xf[((V + 6) >> 1) & 3][V & 1] = cat8(xp[0], xp[1]);
                    }
                    if constexpr (i == 1) {
                        if (4 * w + j >= 1 && (4 * w + j - 1) * SPM < 64) {
#pragma unroll
                            for (int q = 0; q < SPM; ++q) ostore(0, (4 * w + j - 1) * SPM + q, sc, IC<OAUX0>{});
                        }
                    }
                    __builtin_amdgcn_sched_group_barrier(0x008, 1, 0);
                    __builtin_amdgcn_sched_group_barrier(0x100, 4, 0);
                    __builtin_amdgcn_sched_group_barrier(0x020, 1, 0);
                    SB();
                    if (i == 0 && (V & 1) == 0 && j == 0) { cw = *ctrl; SB(); }
                }
            });
        }
        TS(t_end);
        if constexpr (PFIN == 0) {
#pragma unroll
            for (int k = 0; k < 64; ++k) ostore(0, k, sc, IC<OAUX>{});
        } else {
#pragma unroll
            for (int k = (8 * PFIN - 1) * SPM; k < 64; ++k) ostore(0, k, sc, IC<OAUX>{});
        }
#pragma unroll
        for (int k = 0; k < 64; ++k) ostore(1, k, sc, IC<OAUX>{});
    } else if (wave < 6) {
        const int ld = wave - 4;
        const int rr = lane >> 3, p = lane & 7;
        const unsigned voff = rr * NN + ((((p >> 1) ^ ((rr >> 1) & 3)) << 5) | ((p & 1) << 4));
        const v4i rs = make_rsrc(a.w + n0, 4096 * NN - n0);
        auto issue = [&](int s) {
            const unsigned so = (krow(kswap(s, mh), rank) + ld * 64) * NN;
            const unsigned m = L0 + (s & (DEPTH - 1)) * STAGE + ld * 8192;
            __builtin_amdgcn_s_setprio(LPRIO);
#pragma unroll
            for (int q = 0; q < 8; ++q) dma16w(m + q * 1024, voff, rs, so + q * 8 * NN);
            __builtin_amdgcn_s_setprio(0);
        };
        if (ld == 0) __builtin_amdgcn_s_sleep(LDELAY0); else __builtin_amdgcn_s_sleep(LDELAY1);
        issue(0);
        TS(t_iss);
        volatile LAS unsigned* full = (volatile LAS unsigned*)(L + CTRL) + ld;
        volatile LAS v4u* done = (volatile LAS v4u*)(L + CTRL + 16);
        int issued = 1, pub = 0;
        while (pub < NSTEP) {
            if (issued < NSTEP && issued - pub < (pub == 0 ? INFL0 : INFL)) {
                const v4u d = *done;
                const int dm = __builtin_amdgcn_readfirstlane(min(min(d.x, d.y), min(d.z, d.w)));
                const int fr = max(dm + LEAD - HELD, DEPTH - HELD);
                CFENCE();
                if (issued < fr) { issue(issued); ++issued; continue; }
            }
            const int n = issued - pub;
            if (n == 0) { __builtin_amdgcn_s_sleep(2); continue; }
            switch (n) {
                case 1: asm volatile("s_waitcnt vmcnt(0)" ::: "memory"); break;
                case 2: asm volatile("s_waitcnt vmcnt(8)" ::: "memory"); break;
                case 3: asm volatile("s_waitcnt vmcnt(16)" ::: "memory"); break;
                default: asm volatile("s_waitcnt vmcnt(24)" ::: "memory"); break;
            }
            ++pub;
            *full = pub < NSTEP ? (unsigned)pub : BIG;
        }
    } else {
        {
            const unsigned char* px = a.x + (size_t)(rank * ML + mh * 256 + (idx >> 1) * 16) * KSL + lane * 64;
            unsigned t0_, t1_;
            asm volatile("global_load_dword %0, %2, off\n\tglobal_load_dword %1, %3, off\n\ts_waitcnt vmcnt(0)"
                         : "=&v"(t0_), "=&v"(t1_) : "v"(px), "v"(px + 4096) : "memory");
        }
        TS(t_xpf);
#define PF_DELAY 300
        { const u64 t0 = NOW(); while (NOW() - t0 < PF_DELAY) __builtin_amdgcn_s_sleep(8); }
        const unsigned char* wx = a.w + (b & 7) * 2048 + (size_t)(idx * 4) * NN;
        const unsigned l0 = (unsigned)((u64)wx >> 7);
        const unsigned o0 = ((3 - l0) & 7) * 128, o1 = o0 + 1024;
        const unsigned e = (63 - l0) & 63;
        const unsigned o2 = e < 16 ? e * 128 : o1;
        unsigned dmy = 0;
        for (int s = PF_FIRST; s < NSTEP; ++s) {
            const int sq = s < TSW ? s : TSW + ((s - TSW) / (2 * SW2)) * 2 * SW2 + (((s - TSW) % (2 * SW2)) >> 1) + ((s - TSW) & 1) * SW2;
            const unsigned char* p = wx + (size_t)krow(sq, rank) * NN;
            const unsigned char* p0 = p + o0;
            const unsigned char* p1 = p + o1;
#pragma unroll
            for (int q = 0; q < 8; ++q) {
                asm volatile("s_load_dword %0, %1, %2" : "+s"(dmy) : "s"(p0), "i"((q >> 1) * NN + (q & 1) * 64) : "memory");
                asm volatile("s_load_dword %0, %1, %2" : "+s"(dmy) : "s"(p1), "i"((q >> 1) * NN + (q & 1) * 64) : "memory");
            }
            if (e < 16) {
                const unsigned char* p2 = p + o2;
#pragma unroll
                for (int q = 0; q < 8; ++q)
                    asm volatile("s_load_dword %0, %1, %2" : "+s"(dmy) : "s"(p2), "i"((q >> 1) * NN + (q & 1) * 64) : "memory");
            }
        }
        asm volatile("s_waitcnt lgkmcnt(0)" : "+s"(dmy) :: "memory");
    }
}

extern "C" void kernel_launch(int rank, int nranks, ncclComm_t comm,
                              hipStream_t stream,
                              void* const* d_in, const int* in_sizes, int n_in,
                              void* d_out, int out_size,
                              const int* in_dtypes, int out_dtype,
                              void* const* d_out_all,
                              void* const* d_ws_all, size_t ws_bytes) {
    if (nranks != NRANK || (unsigned)rank >= (unsigned)NRANK || n_in < 4 || ws_bytes < WS_NEED
        || out_size != ML * NN || in_sizes[0] != 4096 * KSL || in_sizes[1] != 4096 * NN) {
        fprintf(stderr, "unexpected problem shape\n");
        return;
    }
    static bool init = false;
    if (!init) {
        HIP_CHECK(hipFuncSetAttribute((const void*)a2a_gemm, hipFuncAttributeMaxDynamicSharedMemorySize, LDS_BYTES));
        init = true;
    }
    Args a;
    a.x = (const unsigned char*)d_in[0];
    a.w = (const unsigned char*)d_in[1];
    a.sx = (const float*)d_in[2];
    a.sw = (const float*)d_in[3];
    a.out = (float*)d_out;
    a.rank = rank;
    a.pad = 0;
    for (int j = 0; j < NRANK; ++j) a.ws[j] = (unsigned char*)d_ws_all[(rank - j) & (NRANK - 1)];
    hipLaunchKernelGGL(a2a_gemm, dim3(256), dim3(512), LDS_BYTES, stream, a);
    HIP_CHECK(hipGetLastError());
}
```
